```python
import numpy as np

kernel_name = "gpu_ag_gemm_m4096_k4096_n16384_e4m3e5m2_dequant"


ABSMAX_RTOL = 0.05

COMPUTE_DTYPE = "float32"
EPILOGUE = "dequant"
IN_QUANT = None
OUT_QUANT = None
IN_FP8 = "e4m3e5m2"
IN_DTYPES = {"x": "e4m3", "w_mat": "e5m2",
             "scale_x": "float32", "scale_w": "float32"}
WORLD_SIZE = 8

def _snap_fp8(x, mant_bits, max_abs, min_normal):
    a = np.abs(x.astype(np.float64))
    _m, _e = np.frexp(a)
    q = np.where(a >= min_normal, np.ldexp(1.0, _e - 1 - mant_bits),
                 min_normal * 2.0 ** -mant_bits)
    snapped = np.minimum(np.round(a / q) * q, max_abs)
    return (np.sign(x) * snapped).astype(np.float32)

def setup_inputs():

    x = _snap_fp8(np.random.randn(4096, 4096).astype(np.float32), 3, 448.0, 2.0 ** -6)
    w_mat = _snap_fp8(np.random.randn(4096, 16384).astype(np.float32), 2, 57344.0, 2.0 ** -14)
    scale_x = (0.02 * (1.0 + np.random.rand(1))).astype(np.float32)
    scale_w = (0.02 * (1.0 + np.random.rand(1))).astype(np.float32)
    return {"x": x, "w_mat": w_mat, "scale_x": scale_x, "scale_w": scale_w}

def reference(x, w_mat, scale_x, scale_w):
    acc = (x @ w_mat).astype(np.float32)
    y = (acc * (scale_x * scale_w)).astype(np.float32)
    return y
```

```cpp
#include <hip/hip_runtime.h>
#include <rccl/rccl.h>
#include <cstdio>

#define HIP_CHECK(expr) do { hipError_t _e = (expr); if (_e != hipSuccess) { \
    fprintf(stderr, "HIP error %s at %s:%d\n", hipGetErrorString(_e), __FILE__, __LINE__); \
    return; } } while (0)

namespace {

#define LAS __attribute__((address_space(3)))
#define GAS __attribute__((address_space(1)))
typedef unsigned u32x4 __attribute__((ext_vector_type(4)));
typedef int i32x2 __attribute__((ext_vector_type(2)));
typedef int i32x4 __attribute__((ext_vector_type(4)));
typedef int i32x8 __attribute__((ext_vector_type(8)));
typedef float f32x4 __attribute__((ext_vector_type(4)));

constexpr int NR = 8;
constexpr int KD = 4096;
constexpr int NL = 2048;
constexpr int MS = 512;
constexpr int BM = 128, BN = 256, BK = 128, NT = KD / BK;
constexpr int NWG = 256, NTHR = 512;
constexpr int A_STAGE = BM * BK, B_STAGE = BK * BN, STAGE = A_STAGE + B_STAGE, NSTAGE = 3;
constexpr int LDS_CTRL = NSTAGE * STAGE;
constexpr int LDS_BYTES = LDS_CTRL + 16;

constexpr size_t SHARD_BYTES = (size_t)MS * KD;
constexpr size_t WS_STAGE = 0;
constexpr size_t WS_FLAG = 2 * SHARD_BYTES;
constexpr size_t WS_CNT = WS_FLAG + (size_t)NR * NWG * 4;
constexpr size_t WS_TMO = WS_CNT + (size_t)NWG * 128;
constexpr size_t WS_STAMP = WS_TMO + 128;
constexpr size_t WS_END = WS_STAMP + (size_t)NWG * 64 * 8;
constexpr unsigned long long DEADLINE_TICKS = 50000000ull;

struct Params {
    const unsigned char* x;
    const unsigned char* w;
    const float* sx;
    const float* sw;
    float* out;
    unsigned char* ws[NR];
    int rank;
    int pad;
};

#define WAIT_V(n) asm volatile("s_waitcnt vmcnt(" #n ")" ::: "memory")
#define WAIT_L(n) asm volatile("s_waitcnt lgkmcnt(" #n ")" ::: "memory")
#define BARRIER() do { asm volatile("" ::: "memory"); __builtin_amdgcn_s_barrier(); asm volatile("" ::: "memory"); } while (0)

#define STAMP(i) do {} while (0)

__global__ void __launch_bounds__(NTHR, 2) ag_gemm_kernel(Params p) {
    extern __shared__ __attribute__((aligned(16))) unsigned char smem_raw[];
    LAS unsigned char* lds = (LAS unsigned char*)smem_raw;
    const int tid = threadIdx.x, lane = tid & 63;
    const int wave = __builtin_amdgcn_readfirstlane(tid >> 6);
    const int b = blockIdx.x;
    const int rank = p.rank;
    unsigned char* myws = p.ws[rank];
    STAMP(0);

    unsigned* cntp = (unsigned*)(myws + WS_CNT + (size_t)b * 128);
    const int g = lane >> 4, r16 = lane & 15;
    u32x4 xv;
    {
        const int blk = b >> 1, q = blk >> 5, t = blk & 31, rg = 4 * (b & 1) + (wave >> 1), h = wave & 1;
        const unsigned char* src = p.x + (size_t)(q * BM + rg * 16 + r16) * KD + t * BK + h * 64 + g * 16;
        xv = *(const GAS u32x4*)src;
    }
    const unsigned call = __builtin_amdgcn_readfirstlane(__hip_atomic_load(cntp, __ATOMIC_RELAXED, __HIP_MEMORY_SCOPE_AGENT)) + 1u;
    const size_t slot_off = WS_STAGE + (size_t)(call & 1u) * SHARD_BYTES;
    {
        unsigned char* dst = myws + slot_off + (size_t)b * 8192 + wave * 1024 + lane * 16;
        asm volatile("global_store_dwordx4 %0, %1, off sc0 sc1\n\ts_nop 1" :: "v"(dst), "v"(xv) : "memory");
    }
    WAIT_V(0);
    BARRIER();
    STAMP(1);
    if (wave == 0) {
        if (lane < NR) __hip_atomic_store((unsigned*)(p.ws[lane] + WS_FLAG + ((size_t)rank * NWG + b) * 4), call, __ATOMIC_RELAXED, __HIP_MEMORY_SCOPE_SYSTEM);
        if (lane == 0) __hip_atomic_store(cntp, call, __ATOMIC_RELAXED, __HIP_MEMORY_SCOPE_AGENT);
    }

    const unsigned src_rank = (unsigned)(rank + (b & 7)) & 7u;
    const int idx = b >> 3, panel = idx & 3, ct = idx >> 2;
    const unsigned char* abase = p.ws[src_rank] + slot_off + (size_t)panel * NT * A_STAGE;
    const unsigned char* bbase = p.w + ct * BN;
    const unsigned aoff = (unsigned)(wave * 2048 + lane * 16);
    unsigned boff[4];
#pragma unroll
    for (int j = 0; j < 4; ++j) {
        const int row = 4 * (wave * 4 + j) + g, f = (row & 7) | ((row >> 1) & 8);
        boff[j] = (unsigned)(row * NL + ((r16 ^ f) << 4) - j * 1024);
    }
    const unsigned lds0 = __builtin_amdgcn_readfirstlane((unsigned)(size_t)lds);
#define STAGE_A(t, s) do { unsigned keep_; \
        asm volatile("s_mov_b32 %0, m0\n\ts_mov_b32 m0, %3\n\ts_nop 0\n\t" \
                     "global_load_lds_dwordx4 %1, %2\n\tglobal_load_lds_dwordx4 %1, %2 offset:1024\n\ts_mov_b32 m0, %0" \
                     : "=&s"(keep_) : "v"(aoff), "s"(abase + (size_t)(t) * A_STAGE), "s"(lds0 + (s) * STAGE + wave * 2048) : "memory"); } while (0)
#define STAGE_B(t, s) do { unsigned keep_; \
        asm volatile("s_mov_b32 %0, m0\n\ts_mov_b32 m0, %6\n\ts_nop 0\n\t" \
                     "global_load_lds_dwordx4 %1, %5\n\tglobal_load_lds_dwordx4 %2, %5 offset:1024\n\t" \
                     "global_load_lds_dwordx4 %3, %5 offset:2048\n\tglobal_load_lds_dwordx4 %4, %5 offset:3072\n\ts_mov_b32 m0, %0" \
                     : "=&s"(keep_) : "v"(boff[0]), "v"(boff[1]), "v"(boff[2]), "v"(boff[3]), "s"(bbase + (size_t)(t) * BK * NL), \
                       "s"(lds0 + (s) * STAGE + A_STAGE + wave * 4096) : "memory"); } while (0)

    STAGE_B(0, 0); STAGE_B(1, 1);

    unsigned* tmo = (unsigned*)(myws + WS_TMO);
    volatile LAS unsigned* ctrl = (volatile LAS unsigned*)(lds + LDS_CTRL);
    if (wave == 0) {
        const unsigned* fl = (const unsigned*)(myws + WS_FLAG + ((size_t)src_rank * NWG + panel * 64 + lane) * 4);
        const unsigned long long t0 = __builtin_amdgcn_s_memrealtime();
        unsigned bad = 0;
        while (!__all(__hip_atomic_load(fl, __ATOMIC_RELAXED, __HIP_MEMORY_SCOPE_SYSTEM) >= call)) {
            if (__builtin_amdgcn_readfirstlane(__hip_atomic_load(tmo, __ATOMIC_RELAXED, __HIP_MEMORY_SCOPE_AGENT)) == call
                || __builtin_amdgcn_s_memrealtime() - t0 > DEADLINE_TICKS) {
                if (lane == 0) __hip_atomic_store(tmo, call, __ATOMIC_RELAXED, __HIP_MEMORY_SCOPE_AGENT);
                bad = 1;
                break;
            }
            __builtin_amdgcn_s_sleep(1);
        }
        STAMP(2);
        __builtin_amdgcn_fence(__ATOMIC_ACQUIRE, "");
        WAIT_V(0);
        if (lane == 0) ctrl[0] = bad;
        WAIT_L(0);
    }
    BARRIER();
    STAMP(3);
    const bool bad = __builtin_amdgcn_readfirstlane(ctrl[0]) != 0u;

    f32x4 acc[4][4];
#pragma unroll
    for (int m = 0; m < 4; ++m)
#pragma unroll
        for (int n = 0; n < 4; ++n) acc[m][n] = (f32x4){0.f, 0.f, 0.f, 0.f};
    const int wm = wave >> 2, wn = wave & 3;

    if (!bad) {
        STAGE_A(0, 0); STAGE_A(1, 1);
        const int ard = wm * 8192 + lane * 16;
        const int q8 = r16 >> 1, fl = q8 | ((g & 1) << 3);
        int brd[4];
#pragma unroll
        for (int n = 0; n < 4; ++n) brd[n] = A_STAGE + (16 * g + q8) * 256 + (((wn * 4 + n) ^ fl) << 4) + (lane & 1) * 8;

#define COMPUTE(s) do { \
        i32x8 af[4], bf[4]; \
        _Pragma("unroll") for (int n = 0; n < 4; ++n) { \
            const i32x2 t0_ = __builtin_amdgcn_ds_read_tr8_b64_v2i32((LAS i32x2*)(lds + (s) * STAGE + brd[n])); \
            const i32x2 t1_ = __builtin_amdgcn_ds_read_tr8_b64_v2i32((LAS i32x2*)(lds + (s) * STAGE + brd[n] + 2048)); \
            const i32x2 t2_ = __builtin_amdgcn_ds_read_tr8_b64_v2i32((LAS i32x2*)(lds + (s) * STAGE + brd[n] + 16384)); \
            const i32x2 t3_ = __builtin_amdgcn_ds_read_tr8_b64_v2i32((LAS i32x2*)(lds + (s) * STAGE + brd[n] + 16384 + 2048)); \
            bf[n] = (i32x8){t0_[0], t0_[1], t1_[0], t1_[1], t2_[0], t2_[1], t3_[0], t3_[1]}; } \
        _Pragma("unroll") for (int m = 0; m < 4; ++m) { \
            const i32x4 lo_ = *(const LAS i32x4*)(lds + (s) * STAGE + ard + m * 2048); \
            const i32x4 hi_ = *(const LAS i32x4*)(lds + (s) * STAGE + ard + m * 2048 + 1024); \
            af[m] = (i32x8){lo_[0], lo_[1], lo_[2], lo_[3], hi_[0], hi_[1], hi_[2], hi_[3]}; } \
        _Pragma("unroll") for (int m = 0; m < 4; ++m) \
            _Pragma("unroll") for (int n = 0; n < 4; ++n) \
                acc[m][n] = __builtin_amdgcn_mfma_scale_f32_16x16x128_f8f6f4(bf[n], af[m], acc[m][n], 1, 0, 0, 0, 0, 0); \
        } while (0)

        WAIT_V(2); BARRIER();
        STAMP(4);
        STAGE_A(2, 2); STAGE_B(2, 2);
        COMPUTE(0);
        for (int t = 1; t < NT - 4; t += 3) {
            WAIT_L(0); WAIT_V(6); BARRIER(); STAGE_A(t + 2, 0); STAGE_B(t + 2, 0); COMPUTE(1);
            WAIT_L(0); WAIT_V(6); BARRIER(); STAGE_A(t + 3, 1); STAGE_B(t + 3, 1); COMPUTE(2);
            WAIT_L(0); WAIT_V(6); BARRIER(); STAGE_A(t + 4, 2); STAGE_B(t + 4, 2); COMPUTE(0);
        }
        WAIT_L(0); WAIT_V(6); BARRIER(); STAGE_A(30, 0); STAGE_B(30, 0); COMPUTE(1);
        WAIT_L(0); WAIT_V(6); BARRIER(); STAGE_A(31, 1); STAGE_B(31, 1); COMPUTE(2);
        WAIT_L(0); WAIT_V(6); BARRIER(); COMPUTE(0);
        WAIT_L(0); WAIT_V(0); BARRIER();
        STAMP(5);
        COMPUTE(1);
    } else {
        WAIT_V(0);
    }

    const float scale = bad ? __builtin_nanf("") : p.sx[0] * p.sw[0];
    float* orow = p.out + (size_t)(src_rank * MS + panel * BM + wm * 64 + r16) * NL + ct * BN + wn * 64 + g * 4;
#pragma unroll
    for (int m = 0; m < 4; ++m)
#pragma unroll
        for (int n = 0; n < 4; ++n) {
            const f32x4 v = acc[m][n] * scale;
            *(GAS f32x4*)(orow + (size_t)m * 16 * NL + n * 16) = v;
        }
}

}

extern "C" void kernel_launch(int rank, int nranks, ncclComm_t comm,
                              hipStream_t stream,
                              void* const* d_in, const int* in_sizes, int n_in,
                              void* d_out, int out_size,
                              const int* in_dtypes, int out_dtype,
                              void* const* d_out_all,
                              void* const* d_ws_all, size_t ws_bytes) {
    (void)comm; (void)in_sizes; (void)n_in; (void)out_size; (void)in_dtypes; (void)out_dtype; (void)d_out_all;
    if (nranks != NR || (unsigned)rank >= (unsigned)NR || ws_bytes < WS_END) { fprintf(stderr, "unexpected configuration\n"); return; }
    static bool attr_set = false;
    if (!attr_set) {
        HIP_CHECK(hipFuncSetAttribute((const void*)ag_gemm_kernel, hipFuncAttributeMaxDynamicSharedMemorySize, LDS_BYTES));
        attr_set = true;
    }
    Params p;
    p.x = (const unsigned char*)d_in[0];
    p.w = (const unsigned char*)d_in[1];
    p.sx = (const float*)d_in[2];
    p.sw = (const float*)d_in[3];
    p.out = (float*)d_out;
    for (int i = 0; i < NR; ++i) p.ws[i] = (unsigned char*)d_ws_all[i];
    p.rank = rank;
    p.pad = 0;
    hipLaunchKernelGGL(ag_gemm_kernel, dim3(NWG), dim3(NTHR), LDS_BYTES, stream, p);
    HIP_CHECK(hipGetLastError());
}
```

```cpp
#include <hip/hip_runtime.h>
#include <rccl/rccl.h>
#include <cstdio>

#define HIP_CHECK(expr) do { hipError_t _e = (expr); if (_e != hipSuccess) { \
    fprintf(stderr, "HIP error %s at %s:%d\n", hipGetErrorString(_e), __FILE__, __LINE__); \
    return; } } while (0)

namespace {

#define LAS __attribute__((address_space(3)))
#define GAS __attribute__((address_space(1)))
typedef unsigned u32x4 __attribute__((ext_vector_type(4)));
typedef int i32x2 __attribute__((ext_vector_type(2)));
typedef int i32x4 __attribute__((ext_vector_type(4)));
typedef int i32x8 __attribute__((ext_vector_type(8)));
typedef float f32x4 __attribute__((ext_vector_type(4)));

constexpr int NR = 8;
constexpr int KD = 4096;
constexpr int NL = 2048;
constexpr int MS = 512;
constexpr int BM = 128, BN = 256, BK = 128, NT = KD / BK;
constexpr int NWG = 256, NTHR = 512;
constexpr int A_STAGE = BM * BK, B_STAGE = BK * BN;
constexpr int LDS_A = 0, LDS_B = 2 * A_STAGE, LDS_BYTES = LDS_B + 4 * B_STAGE;

constexpr size_t STAGE_SLOT = (size_t)4 * NT * A_STAGE;
constexpr size_t WS_STAGE = 0;
constexpr size_t WS_FLAG1 = WS_STAGE + 2 * STAGE_SLOT;
constexpr size_t WS_CNT = WS_FLAG1 + (size_t)NR * NWG * 4;
constexpr size_t WS_TMO = WS_CNT + (size_t)NWG * 128;
constexpr size_t WS_STAMP = WS_TMO + 128;
constexpr size_t WS_END = WS_STAMP + (size_t)NWG * 64 * 8;
constexpr unsigned long long DEADLINE_TICKS = 50000000ull;
#define AG_PRE 20
#define AG_HOLD 150

#define AG_INV 200
#define AG_PH 22

#define AG_PFW 1
#define AG_PFL 32

struct Params {
    const unsigned char* x;
    const unsigned char* w;
    const float* sx;
    const float* sw;
    float* out;
    unsigned char* wsr[NR];
    int rank;
    unsigned srcs;
};

#define WAIT_V(n) asm volatile("s_waitcnt vmcnt(" #n ")" ::: "memory")
#define WAIT_L(n) asm volatile("s_waitcnt lgkmcnt(" #n ")" ::: "memory")
#define BARRIER() do { asm volatile("" ::: "memory"); __builtin_amdgcn_s_barrier(); asm volatile("" ::: "memory"); } while (0)
#define SCHED() __builtin_amdgcn_sched_barrier(0)

#define STAMPC(i) do {} while (0)
#define STAMP(i) do {} while (0)

__global__ void __launch_bounds__(NTHR, 2) ag_gemm_kernel(Params p) {
    extern __shared__ __attribute__((aligned(16))) unsigned char smem_raw[];
    LAS unsigned char* lds = (LAS unsigned char*)smem_raw;
    const int tid = threadIdx.x, lane = tid & 63;
    const int wave = __builtin_amdgcn_readfirstlane(tid >> 6) & 7;
    const int b = blockIdx.x;
    const int grp = b & 7, idx = b >> 3;
    int rank = p.rank;
    unsigned srcs = p.srcs;
    unsigned char* myws = p.wsr[0];
    const unsigned char* srcws = p.wsr[grp];
    const unsigned char* xp = p.x;
    const unsigned char* wp = p.w;
    const float* sxp = p.sx;
    const float* swp = p.sw;
    float* outp = p.out;
    asm volatile("" : "+s"(rank), "+s"(srcs), "+s"(myws), "+s"(srcws), "+s"(xp), "+s"(wp), "+s"(sxp), "+s"(swp), "+s"(outp));
    STAMP(0);

    const int g = lane >> 4, r16 = lane & 15;
    const unsigned src_rank = (srcs >> (4 * grp)) & 7u;
    const int panel = idx & 3, ct = idx >> 2;
    const unsigned char* bbase = wp + ct * BN;
#define KS(t) (t)
    const unsigned aoff = (unsigned)(wave * 2048 + lane * 16);
    unsigned boff[4];
#pragma unroll
    for (int j = 0; j < 4; ++j) {
        const int row = 4 * (wave * 4 + j) + g, f = (row & 7) | ((row >> 1) & 8);
        boff[j] = (unsigned)(row * NL + ((r16 ^ f) << 4) - j * 1024);
    }
    const unsigned lds0 = __builtin_amdgcn_readfirstlane((unsigned)(size_t)lds);
#define STAGE_B4(off, src, dst) do { unsigned keep_; \
        asm volatile("s_mov_b32 %0, m0\n\ts_mov_b32 m0, %6\n\ts_nop 0\n\t" \
                     "global_load_lds_dwordx4 %1, %5\n\tglobal_load_lds_dwordx4 %2, %5 offset:1024\n\t" \
                     "global_load_lds_dwordx4 %3, %5 offset:2048\n\tglobal_load_lds_dwordx4 %4, %5 offset:3072\n\ts_mov_b32 m0, %0" \
                     : "=&s"(keep_) : "v"(off[0]), "v"(off[1]), "v"(off[2]), "v"(off[3]), "s"(src), "s"(dst) : "memory"); } while (0)
#define STAGE_B(t) STAGE_B4(boff, bbase + (size_t)KS(t) * (BK * NL), lds0 + LDS_B + ((t) & 3) * B_STAGE + wave * 4096)
#define STAGE_B1(j, src, dst) do { unsigned keep_; \
        asm volatile("s_mov_b32 %0, m0\n\ts_mov_b32 m0, %3\n\ts_nop 0\n\tglobal_load_lds_dwordx4 %1, %2 offset:%4\n\ts_mov_b32 m0, %0" \
                     : "=&s"(keep_) : "v"(boff[j]), "s"(src), "s"(dst), "i"((j) * 1024) : "memory"); } while (0)
#define LOAD_A0(t, r) asm volatile("global_load_dwordx4 %0, %2, %3\n\tglobal_load_dwordx4 %1, %2, %3 offset:1024" \
                                   : "=&v"(r[0]), "=&v"(r[1]) : "v"(aoff), "s"(apull + (size_t)KS(t) * A_STAGE) : "memory")
#define LOAD_A(t, r) do { LOAD_A0(t, r); \
        if (AG_PFW > 0 && (t) + 1 < NT && wave >= 8 - AG_PFW && (((t) ^ ct) & 7) == 0) \
            asm volatile("global_load_dword %0, %1, %2" : "+v"(pf) : "v"(pfoff), "s"(apull + (size_t)((t) + 1) * A_STAGE) : "memory"); } while (0)
    unsigned pf = 0;
    const unsigned pfoff = (unsigned)(((7 - wave) * AG_PFL + (lane & (AG_PFL - 1))) * 128);

    GAS unsigned* cntp = (GAS unsigned*)(myws + WS_CNT + (size_t)b * 128);
    u32x4 xv[2];
    unsigned cnt; float sx, sw;
    asm volatile("global_load_dword %0, %3, %4 sc1\n\tglobal_load_dword %1, %3, %5\n\tglobal_load_dword %2, %3, %6"
                 : "=&v"(cnt), "=&v"(sx), "=&v"(sw) : "v"(0u), "s"(cntp), "s"(sxp), "s"(swp) : "memory");
    unsigned char* peer = nullptr;
    if (wave < 4) {
        peer = p.wsr[lane & 7];
        const int q = b >> 6, j = (b >> 1) & 31;
        const int t = j < NT - 4 ? j : NT - 4 + wave;
        const int rg = (b & 1) * 4 + (j < NT - 4 ? wave : j - (NT - 4));
        asm volatile("global_load_dwordx4 %0, %2, %3\n\tglobal_load_dwordx4 %1, %2, %3 offset:64\n\ts_waitcnt vmcnt(0)"
                     : "=&v"(xv[0]), "=&v"(xv[1]) : "v"((unsigned)(r16 * KD + g * 16)), "s"(xp + (size_t)(q * BM + rg * 16) * KD + t * BK) : "memory");
        asm volatile("" : "+v"(peer));
    } else {
        WAIT_V(0);
    }
    asm volatile("" : "+v"(cnt), "+v"(sx), "+v"(sw));
    const unsigned call = __builtin_amdgcn_readfirstlane(cnt) + 1u;
    const unsigned scale0 = __builtin_amdgcn_readfirstlane(__builtin_bit_cast(unsigned, sx * sw));
    const size_t stage_off = WS_STAGE + (size_t)(call & 1u) * STAGE_SLOT;
    const unsigned char* apull = srcws + stage_off + (size_t)panel * NT * A_STAGE;
    if (wave < 4) {
        asm volatile("global_store_dwordx4 %0, %1, %3 sc0 sc1\n\tglobal_store_dwordx4 %0, %2, %3 offset:1024 sc0 sc1\n\ts_nop 1"
                     :: "v"(aoff), "v"(xv[0]), "v"(xv[1]), "s"(myws + stage_off + (size_t)b * 8192) : "memory");
        WAIT_V(0);
    }
    BARRIER();
    if (wave < 4) {
        if (wave == 1 && lane < NR) __hip_atomic_store((GAS unsigned*)(peer + WS_FLAG1 + ((size_t)rank * NWG + b) * 4), call, __ATOMIC_RELAXED, __HIP_MEMORY_SCOPE_SYSTEM);
        if (wave == 2 && lane == 0) __hip_atomic_store(cntp, call, __ATOMIC_RELAXED, __HIP_MEMORY_SCOPE_AGENT);
    } else {
        __builtin_amdgcn_s_sleep(1);
        STAGE_B(0); STAGE_B4(boff, bbase + (KS(0) * BK - 64) * NL, lds0 + LDS_B + (wave - 4) * 4096);
        STAGE_B(1); STAGE_B4(boff, bbase + (KS(1) * BK - 64) * NL, lds0 + LDS_B + B_STAGE + (wave - 4) * 4096);
        STAGE_B(2); STAGE_B4(boff, bbase + (KS(2) * BK - 64) * NL, lds0 + LDS_B + 2 * B_STAGE + (wave - 4) * 4096);
        WAIT_V(0);
    }
    STAMP(1);

    GAS unsigned* tmo = (GAS unsigned*)(myws + WS_TMO);
    volatile LAS unsigned* ctrl = (volatile LAS unsigned*)(lds + LDS_A + A_STAGE);
    unsigned long long t1 = 0;
    if (wave != 0) {
        BARRIER();
    } else {
        const unsigned char* fl = myws + WS_FLAG1 + (size_t)src_rank * NWG * 4 + lane * 16;
        const unsigned long long t0 = __builtin_amdgcn_s_memrealtime();
        unsigned bad = 0;
        u32x4 f0, f1;
#define POLL(f) asm volatile("global_load_dwordx4 %0, %1, off sc0 sc1" : "=&v"(f) : "v"(fl) : "memory")
#define SEEN(f) __all(f[0] >= call && f[1] >= call && f[2] >= call && f[3] >= call)
        POLL(f0);
        for (unsigned it = 1;; ++it) {
            __builtin_amdgcn_s_sleep(2);
            POLL(f1); asm volatile("s_waitcnt vmcnt(1)" : "+v"(f0) :: "memory");
            if (SEEN(f0)) break;
            __builtin_amdgcn_s_sleep(2);
            POLL(f0); asm volatile("s_waitcnt vmcnt(1)" : "+v"(f1) :: "memory");
            if (SEEN(f1)) break;
            if ((it & 63u) == 0u
                && (__builtin_amdgcn_readfirstlane(__hip_atomic_load(tmo, __ATOMIC_RELAXED, __HIP_MEMORY_SCOPE_AGENT)) == call
                    || __builtin_amdgcn_s_memrealtime() - t0 > DEADLINE_TICKS)) {
                if (lane == 0) __hip_atomic_store(tmo, call, __ATOMIC_RELAXED, __HIP_MEMORY_SCOPE_AGENT);
                bad = 1;
                break;
            }
        }
        BARRIER();
        t1 = __builtin_amdgcn_s_memrealtime();
        asm volatile("s_waitcnt vmcnt(0)" : "+v"(f0), "+v"(f1) :: "memory");
        STAMP(2);
        while (__builtin_amdgcn_s_memrealtime() - t1 < (unsigned long long)AG_PRE) {}
        __builtin_amdgcn_fence(__ATOMIC_ACQUIRE, "");
        WAIT_V(0);
        STAMP(3);
        if (lane == 0) ctrl[0] = bad;
        while (__builtin_amdgcn_s_memrealtime() - t1 < (unsigned long long)(panel == 0 ? AG_HOLD : AG_INV + panel * AG_PH)) __builtin_amdgcn_s_sleep(1);
        WAIT_L(0);
    }
    BARRIER();
    STAMP(4);
    const bool bad = __builtin_amdgcn_readfirstlane(ctrl[0]) != 0u;
    const int wm = wave >> 2, wn = wave & 3;
    float* orow = outp + (size_t)(src_rank * MS + panel * BM + wm * 64 + (lane >> 3)) * NL + ct * BN + wn * 64 + (lane & 7) * 4;
#define STORE_OUT1(m, i, np, v) do { const f32x4 o_ = (v); \
        asm volatile("global_store_dwordx4 %0, %1, off offset:%2 sc1\n\ts_nop 1" :: "v"(orow + (size_t)((m) * 16 + (i) * 8) * NL), "v"(o_), "i"((np) * 128) : "memory"); } while (0)

    if (bad) {
        const float q = __builtin_nanf("");
#pragma unroll
        for (int m = 0; m < 4; ++m)
#pragma unroll
            for (int n = 0; n < 4; ++n) STORE_OUT1(m, n & 1, n >> 1, ((f32x4){q, q, q, q}));
        return;
    }

    u32x4 a0[2], a1[2];
    unsigned tmov = 0;
    asm volatile("global_load_dword %0, %1, off sc1" : "+v"(tmov) : "v"(tmo) : "memory");
    LOAD_A(0, a0);
    STAGE_B(3);

    f32x4 acc[4][4];
#pragma unroll
    for (int m = 0; m < 4; ++m)
#pragma unroll
        for (int n = 0; n < 4; ++n) acc[m][n] = (f32x4){0.f, 0.f, 0.f, 0.f};
    const int awr = LDS_A + wave * 2048 + lane * 16;
    const int ard = LDS_A + wm * 8192 + lane * 16;
    unsigned brd = (16 * g + (r16 >> 1)) * 256 + (((wn * 4) ^ ((r16 >> 1) | ((g & 1) << 3))) << 4) + (lane & 1) * 8;
#define READ_B1(dst, n, so) do { \
        asm volatile("" : "+v"(brd));                  \
        LAS unsigned char* bp_ = (LAS unsigned char*)(size_t)((brd ^ ((n) << 4)) + (lds0 + LDS_B + (so))); \
        const i32x2 t0_ = __builtin_amdgcn_ds_read_tr8_b64_v2i32((LAS i32x2*)bp_); \
        const i32x2 t1_ = __builtin_amdgcn_ds_read_tr8_b64_v2i32((LAS i32x2*)(bp_ + 2048)); \
        const i32x2 t2_ = __builtin_amdgcn_ds_read_tr8_b64_v2i32((LAS i32x2*)(bp_ + 16384)); \
        const i32x2 t3_ = __builtin_amdgcn_ds_read_tr8_b64_v2i32((LAS i32x2*)(bp_ + 16384 + 2048)); \
        dst[n] = (i32x8){t0_[0], t0_[1], t1_[0], t1_[1], t2_[0], t2_[1], t3_[0], t3_[1]}; } while (0)
#define READ_B(dst, t) do { const int so_ = ((t) & 3) * B_STAGE; READ_B1(dst, 0, so_); READ_B1(dst, 1, so_); READ_B1(dst, 2, so_); READ_B1(dst, 3, so_); } while (0)
#define READ_A1(dst, slot, m) do { \
        const i32x4 lo_ = *(const LAS i32x4*)(lds + (slot) * A_STAGE + ard + (m) * 2048); \
        const i32x4 hi_ = *(const LAS i32x4*)(lds + (slot) * A_STAGE + ard + (m) * 2048 + 1024); \
        dst = (i32x8){lo_[0], lo_[1], lo_[2], lo_[3], hi_[0], hi_[1], hi_[2], hi_[3]}; } while (0)
#define READ_A(dst, slot) do { READ_A1(dst[0], slot, 0); READ_A1(dst[1], slot, 1); READ_A1(dst[2], slot, 2); READ_A1(dst[3], slot, 3); } while (0)
#define PUT_A(r, slot) do { asm volatile("" : "+v"(r[0]), "+v"(r[1])); \
        *(LAS u32x4*)(lds + (slot) * A_STAGE + awr) = r[0]; *(LAS u32x4*)(lds + (slot) * A_STAGE + awr + 1024) = r[1]; } while (0)
#define MFMA(c, bfr, afr) c = __builtin_amdgcn_mfma_scale_f32_16x16x128_f8f6f4(bfr, afr, c, 1, 0, 0, 0, 0, 0)
#define MMA_ROW(m, a, bf) do { MFMA(acc[m][0], bf[0], a); MFMA(acc[m][1], bf[1], a); MFMA(acc[m][2], bf[2], a); MFMA(acc[m][3], bf[3], a); } while (0)
#define BODY(t, slot, bprev, bcur, bnxt, tnxt, a3cur, a3prev, FIRST, DMA, MID1, MID2) do { \
        i32x8 f0_, f1_, f2_; READ_A1(f0_, slot, 0); READ_A1(f1_, slot, 1); READ_A1(f2_, slot, 2); READ_A1(a3cur, slot, 3); \
        const int so_ = ((tnxt) & 3) * B_STAGE; \
        const unsigned char* bs_ = bbase + (size_t)KS((t) + 4) * (BK * NL); \
        const unsigned bd_ = lds0 + LDS_B + ((t) & 3) * B_STAGE + wave * 4096; \
        SCHED(); \
        if (DMA) STAGE_B1(0, bs_, bd_); \
        SCHED(); \
        if (!(FIRST)) MMA_ROW(3, a3prev, bprev); \
        SCHED(); if (DMA) STAGE_B1(1, bs_, bd_); SCHED(); \
        MMA_ROW(0, f0_, bcur); SCHED(); READ_B1(bnxt, 0, so_); READ_B1(bnxt, 1, so_); if (DMA) STAGE_B1(2, bs_, bd_); SCHED(); \
        MMA_ROW(1, f1_, bcur); SCHED(); READ_B1(bnxt, 2, so_); READ_B1(bnxt, 3, so_); if (DMA) STAGE_B1(3, bs_, bd_); MID1; SCHED(); \
        MMA_ROW(2, f2_, bcur); SCHED(); WAIT_L(0); MID2; BARRIER(); STAMP(16 + (t)); } while (0)

    i32x8 bf0[4], bf1[4], bf2[4], bf3[4], a3e, a3o;
    READ_B(bf0, 0);
    WAIT_V(4); LOAD_A(1, a1); PUT_A(a0, 0); WAIT_L(0); BARRIER();
    STAMP(6);
    asm volatile("" : "+v"(tmov));
    unsigned scale_bits = __builtin_amdgcn_readfirstlane(tmov) == call ? 0x7fc00000u : scale0;
    asm volatile("" : "+s"(scale_bits));
    const float scale = __builtin_bit_cast(float, scale_bits);
    BODY(0, 0, bf1, bf0, bf1, 1, a3e, a3o, true, true, { WAIT_V(4); LOAD_A(2, a0); PUT_A(a1, 1); }, {});
    for (int t = 1; t < NT - 7; t += 2) {
        BODY(t, 1, bf0, bf1, bf0, t + 1, a3o, a3e, false, true, { WAIT_V(4); LOAD_A(t + 2, a1); PUT_A(a0, 0); }, {});
        BODY(t + 1, 0, bf1, bf0, bf1, t + 2, a3e, a3o, false, true, { WAIT_V(4); LOAD_A(t + 3, a0); PUT_A(a1, 1); }, {});
    }
    BODY(25, 1, bf0, bf1, bf0, 26, a3o, a3e, false, true, { READ_B(bf2, 27); WAIT_V(4); LOAD_A(27, a1); PUT_A(a0, 0); }, {});
    BODY(26, 0, bf1, bf0, bf1, 28, a3e, a3o, false, true, { WAIT_V(4); PUT_A(a1, 1); STAGE_B(31); LOAD_A(28, a0); }, {});
    BODY(27, 1, bf0, bf2, bf0, 29, a3o, a3e, false, false, { MMA_ROW(3, a3o, bf2); SCHED(); },
         { WAIT_V(0); LOAD_A(29, a1); PUT_A(a0, 0); WAIT_L(0); });
    LAS unsigned char* twr = lds + LDS_B + wave * 4096 + r16 * 128;
    const LAS unsigned char* trd = lds + LDS_B + wave * 4096 + (lane >> 3) * 128 + (((lane & 7) ^ (lane >> 3)) << 4);
#define TAIL_COL(m, n, af) do { MFMA(acc[m][n], bf1[n], af[0]); MFMA(acc[m][n], bf0[n], af[1]); MFMA(acc[m][n], bf3[n], af[2]); MFMA(acc[m][n], bf2[n], af[3]); } while (0)
#define TAIL_PUT(m, n) *(LAS f32x4*)(twr + ((n) >> 1) * 2048 + (((((n) & 1) * 4 + g) ^ (r16 & 7)) << 4)) = acc[m][n] * scale
#define TAIL_OUT(m, np) do { const f32x4 r0_ = *(const LAS f32x4*)(trd + (np) * 2048), r1_ = *(const LAS f32x4*)(trd + (np) * 2048 + 1024); \
        STORE_OUT1(m, 0, np, r0_); STORE_OUT1(m, 1, np, r1_); } while (0)
#define TAIL_MMA(m, af) do { \
        TAIL_COL(m, 0, af); SCHED(); TAIL_COL(m, 1, af); TAIL_PUT(m, 0); SCHED(); \
        TAIL_COL(m, 2, af); TAIL_PUT(m, 1); SCHED(); \
        TAIL_COL(m, 3, af); TAIL_PUT(m, 2); TAIL_OUT(m, 0); SCHED(); \
        TAIL_PUT(m, 3); TAIL_OUT(m, 1); } while (0)
    { i32x8 af[4]; READ_A(af, 0); READ_B(bf3, 30); READ_B(bf2, 31); SCHED(); TAIL_MMA(0, af); }
    WAIT_V(4); LOAD_A(30, a0); PUT_A(a1, 1); WAIT_L(0); BARRIER();
    STAMP(8);
    { i32x8 af[4]; READ_A(af, 1); SCHED(); TAIL_MMA(1, af); }
    WAIT_V(4); LOAD_A(31, a1); PUT_A(a0, 0); WAIT_L(0); BARRIER();
    STAMP(9);
    { i32x8 af[4]; READ_A(af, 0); SCHED(); TAIL_MMA(2, af); }
    WAIT_V(4); asm volatile("" : "+v"(pf)); PUT_A(a1, 1); WAIT_L(0); BARRIER();
    STAMP(10);
    { i32x8 af[4]; READ_A(af, 1); SCHED(); TAIL_MMA(3, af); }
}

}

extern "C" void kernel_launch(int rank, int nranks, ncclComm_t comm,
                              hipStream_t stream,
                              void* const* d_in, const int* in_sizes, int n_in,
                              void* d_out, int out_size,
                              const int* in_dtypes, int out_dtype,
                              void* const* d_out_all,
                              void* const* d_ws_all, size_t ws_bytes) {
    (void)comm; (void)in_sizes; (void)n_in; (void)out_size; (void)in_dtypes; (void)out_dtype; (void)d_out_all;
    if (nranks != NR || (unsigned)rank >= (unsigned)NR || ws_bytes < WS_END) { fprintf(stderr, "unexpected configuration\n"); return; }
    static bool attr_set = false;
    if (!attr_set) {
        HIP_CHECK(hipFuncSetAttribute((const void*)ag_gemm_kernel, hipFuncAttributeMaxDynamicSharedMemorySize, LDS_BYTES));
        attr_set = true;
    }
    static const int side[NR] = {0, 1, 0, 0, 1, 1, 0, 1};
    Params p;
    p.x = (const unsigned char*)d_in[0];
    p.w = (const unsigned char*)d_in[1];
    p.sx = (const float*)d_in[2];
    p.sw = (const float*)d_in[3];
    p.out = (float*)d_out;
    int order[NR], n = 0;
    order[n++] = rank;
    for (int r = 0; r < NR; ++r) if (r != rank && side[r] == side[rank]) order[n++] = r;
    for (int r = 0; r < NR; ++r) if (side[r] != side[rank]) order[n++] = r;
    p.srcs = 0;
    for (int i = 0; i < NR; ++i) { p.wsr[i] = (unsigned char*)d_ws_all[order[i]]; p.srcs |= (unsigned)order[i] << (4 * i); }
    p.rank = rank;
    hipLaunchKernelGGL(ag_gemm_kernel, dim3(NWG), dim3(NTHR), LDS_BYTES, stream, p);
    HIP_CHECK(hipGetLastError());
}
```

```cpp
#include <hip/hip_runtime.h>
#include <rccl/rccl.h>
#include <cstdio>

#define HIP_CHECK(expr) do { hipError_t _e = (expr); if (_e != hipSuccess) { \
    fprintf(stderr, "HIP error %s at %s:%d\n", hipGetErrorString(_e), __FILE__, __LINE__); \
    return; } } while (0)

namespace {

#define LAS __attribute__((address_space(3)))
#define GAS __attribute__((address_space(1)))
typedef unsigned u32x4 __attribute__((ext_vector_type(4)));
typedef unsigned u32x2 __attribute__((ext_vector_type(2)));
typedef unsigned u32x3 __attribute__((ext_vector_type(3)));
typedef int i32x2 __attribute__((ext_vector_type(2)));
typedef int i32x4 __attribute__((ext_vector_type(4)));
typedef int i32x8 __attribute__((ext_vector_type(8)));
typedef float f32x4 __attribute__((ext_vector_type(4)));

constexpr int NR = 8;
constexpr int KD = 4096;
constexpr int NL = 2048;
constexpr int MS = 512;
constexpr int BM = 128, BN = 256, BK = 128, NT = KD / BK;
constexpr int NWG = 256, NTHR = 512;
constexpr int NBLK = 4 * NT;
constexpr int A_STAGE = BM * BK, B_STAGE = BK * BN;
constexpr int LDS_A = 0, LDS_B = 2 * A_STAGE, LDS_BYTES = LDS_B + 4 * B_STAGE;

constexpr int ESC_CAP = 39, ESC_BYTES = 2 * (ESC_CAP + 1);
constexpr int CB_D1 = 8 * 1024, CB_ESC = CB_D1 + 8 * 768, CBLK = CB_ESC + 8 * ESC_BYTES;
static_assert(CBLK % 128 == 0, "packed blocks start on a cache line");

constexpr size_t PLANES = (size_t)4 * NT * CBLK;
constexpr size_t STAGE_SLOT = PLANES + (size_t)4 * NT * 8 * 256;
constexpr size_t WS_STAGE = 0;
constexpr size_t WS_FLAG1 = WS_STAGE + 2 * STAGE_SLOT;
constexpr size_t WS_FLAG2 = WS_FLAG1 + (size_t)NR * 8192;
constexpr size_t WS_PRE = WS_FLAG2 + (size_t)NR * 4096;
constexpr size_t WS_CNT = WS_PRE + (size_t)NR * 8 * CBLK;
constexpr size_t WS_TMO = WS_CNT + (size_t)NWG * 128;
constexpr size_t WS_STAMP = WS_TMO + 128;
constexpr size_t WS_END = WS_STAMP + (size_t)NWG * 64 * 8;
constexpr unsigned long long DEADLINE_TICKS = 50000000ull;
#define AG_PRE 20
#define AG_HOLD 150

#define AG_INV 200
#define AG_PH 8

#define AG_TW 0

#define AG_BDLY 3

#define AG_DUP 0

#define AG_NPRE 1
#define AG_TP0 11
#define AG_TPS 6

#define AG_PFS 0

#define AG_TPQ 3

struct Params {
    const unsigned char* x;
    const unsigned char* w;
    const float* sx;
    const float* sw;
    float* out;
    unsigned char* wsr[NR];
    int rank;
    unsigned srcs;
};

#define WAIT_V(n) asm volatile("s_waitcnt vmcnt(" #n ")" ::: "memory")
#define WAIT_L(n) asm volatile("s_waitcnt lgkmcnt(" #n ")" ::: "memory")
#define BARRIER() do { asm volatile("" ::: "memory"); __builtin_amdgcn_s_barrier(); asm volatile("" ::: "memory"); } while (0)
#define SCHED() __builtin_amdgcn_sched_barrier(0)

#define STAMP(i) do {} while (0)

#define STAMPL(t) do {} while (0)
#define FS(on, i) do {} while (0)
#define FS0(i) do {} while (0)

__device__ __forceinline__ unsigned wave_scan(unsigned v) {
    v += __builtin_amdgcn_update_dpp(0, (int)v, 0x111, 0xf, 0xf, true);
    v += __builtin_amdgcn_update_dpp(0, (int)v, 0x112, 0xf, 0xf, true);
    v += __builtin_amdgcn_update_dpp(0, (int)v, 0x114, 0xf, 0xf, true);
    v += __builtin_amdgcn_update_dpp(0, (int)v, 0x118, 0xf, 0xf, true);
    v += __builtin_amdgcn_update_dpp(0, (int)v, 0x142, 0xa, 0xf, false);
    v += __builtin_amdgcn_update_dpp(0, (int)v, 0x143, 0xc, 0xf, false);
    return v;
}

__global__ void __launch_bounds__(NTHR, 2) ag_gemm_kernel(Params p) {
    extern __shared__ __attribute__((aligned(16))) unsigned char smem_raw[];
    LAS unsigned char* lds = (LAS unsigned char*)smem_raw;
    FS0(0);
    const int tid = threadIdx.x, lane = tid & 63;
    const int wave = __builtin_amdgcn_readfirstlane(tid >> 6) & 7;
    const int b = blockIdx.x;
    unsigned lane16 = (unsigned)lane * 16u;
    const int grp = b & 7, idx = b >> 3;
    int rank = p.rank;
    unsigned srcs = p.srcs;
    unsigned char* myws = p.wsr[0];
    const unsigned char* srcws = p.wsr[grp];
    const unsigned char* wavews = p.wsr[idx >> 2];
    const unsigned char* xp = p.x;
    const unsigned char* wp = p.w;
    const float* sxp = p.sx;
    const float* swp = p.sw;
    float* outp = p.out;
    const int g = lane >> 4, r16 = lane & 15;
    const int blk = (b >> 3) * 4 + (b & 3);
    unsigned xoff, cntoff = (unsigned)(WS_CNT + (size_t)b * 128), flagoff = (unsigned)(WS_FLAG1 + (blk & 63) * 128 + (blk >> 6) * 8 + (AG_DUP ? 0 : (b & 4)));
    const bool stager = AG_DUP || ((wave ^ b) & 4) == 0;
    {
        const int q = blk >> 5, j = blk & 31;
        const int t = j < NT - 4 ? j : NT - 4 + (wave & 3);
        const int rg = j < NT - 4 ? wave : (wave & 4) + j - (NT - 4);
        xoff = (unsigned)((q * BM + rg * 16 + (lane >> 3)) * KD + t * BK + (((lane & 7) ^ (lane >> 4)) << 4));
    }
    unsigned xoff8 = (xoff ^ 64u) + 8 * KD - 1024;
    const unsigned lds0 = __builtin_amdgcn_readfirstlane((unsigned)(size_t)lds);
    asm volatile("" : "+v"(xoff), "+v"(xoff8), "+v"(cntoff), "+v"(flagoff));
    asm volatile("" : "+s"(rank), "+s"(srcs), "+s"(myws), "+s"(srcws), "+s"(wavews), "+s"(xp), "+s"(wp), "+s"(sxp), "+s"(swp), "+s"(outp));
    FS0(1);
    unsigned cnt, done = 0; float sx, sw;
    if (stager)
        asm volatile("s_mov_b32 m0, %3\n\ts_nop 0\n\tglobal_load_lds_dwordx4 %0, %2\n\tglobal_load_lds_dwordx4 %1, %2 offset:1024"
                     :: "v"(xoff), "v"(xoff8), "s"(xp), "s"(lds0 + LDS_A + wave * 2048) : "memory");
    asm volatile("global_load_dword %0, %3, %4 sc1\n\tglobal_load_dword %1, %5, %6\n\tglobal_load_dword %2, %5, %7"
                 : "=&v"(cnt), "=&v"(sx), "=&v"(sw) : "v"(cntoff), "s"(myws), "v"(0u), "s"(sxp), "s"(swp) : "memory");
    flagoff += (unsigned)rank * 8192;
    if (AG_DUP) asm volatile("global_load_dword %0, %1, %2 sc0 sc1" : "+v"(done) : "v"(flagoff), "s"(myws) : "memory");
    unsigned char* peer = p.wsr[lane & 7];
    STAMP(0);

    const unsigned src_rank = (srcs >> (4 * grp)) & 7u;
    const int panel = idx & 3, ct = idx >> 2;
    const unsigned char* bbase = wp + ct * BN;
    const unsigned aoff1 = (unsigned)(CB_D1 - wave * 256 + lane * 12);
    const unsigned aoffe = (unsigned)(CB_ESC + wave * (ESC_BYTES - 1024) + (lane <= ESC_CAP ? lane : 0) * 2);
    unsigned boff[4];
#pragma unroll
    for (int j = 0; j < 4; ++j) {
        const int row = 4 * (wave * 4 + j) + g, f = (row & 7) | ((row >> 1) & 8);
        boff[j] = (unsigned)(row * NL + ((r16 ^ f) << 4) - j * 1024);
    }
#define STAGE_B4(off, src, dst) \
        asm volatile("s_mov_b32 m0, %5\n\ts_nop 0\n\t" \
                     "global_load_lds_dwordx4 %0, %4\n\tglobal_load_lds_dwordx4 %1, %4 offset:1024\n\t" \
                     "global_load_lds_dwordx4 %2, %4 offset:2048\n\tglobal_load_lds_dwordx4 %3, %4 offset:3072" \
                     :: "v"(off[0]), "v"(off[1]), "v"(off[2]), "v"(off[3]), "s"(src), "s"(dst) : "memory")
#define STAGE_B(t) STAGE_B4(boff, bbase + (size_t)(t) * (BK * NL), lds0 + LDS_B + ((t) & 3) * B_STAGE + wave * 4096)
#define STAGE_B1(j, src, dst) \
        asm volatile("s_mov_b32 m0, %2\n\ts_nop 0\n\tglobal_load_lds_dwordx4 %0, %1 offset:%3" \
                     :: "v"(boff[j]), "s"(src), "s"(dst), "i"((j) * 1024) : "memory")
#define LOAD_A(t, r) do { \
        if (__builtin_expect((lmask >> (t)) & 1u, 0)) \
            asm volatile("global_load_ushort %2, %5, %6 sc0 sc1\n\tglobal_load_dwordx4 %0, %3, %6 sc0 sc1\n\tglobal_load_dwordx3 %1, %4, %6 sc0 sc1" \
                         : "=&v"(r.d0), "=&v"(r.d1), "=&v"(r.e) : "v"(lane16), "v"(aoff1), "v"(aoffe), \
                           "s"(grp == 0 ? apull + (size_t)(t) * CBLK : apre + ((t) > AG_TP0 + AG_TPQ * panel ? 4 * CBLK : 0)) : "memory"); \
        else \
            asm volatile("global_load_ushort %2, %5, %6\n\tglobal_load_dwordx4 %0, %3, %6\n\tglobal_load_dwordx3 %1, %4, %6" \
                         : "=&v"(r.d0), "=&v"(r.d1), "=&v"(r.e) : "v"(lane16), "v"(aoff1), "v"(aoffe), "s"(apull + (size_t)(t) * CBLK) : "memory"); \
        } while (0)
    const unsigned lmask = grp == 0 ? 0xffffffffu : (AG_NPRE > 0 ? 1u << (AG_TP0 + AG_TPQ * panel) : 0u) | (AG_NPRE > 1 ? 1u << (AG_TP0 + AG_TPS + AG_TPQ * panel) : 0u);
    const unsigned char* apre = myws + WS_PRE + (size_t)(grp * 8 + panel) * CBLK + wave * 1024;

    peer += flagoff;
    asm volatile("s_waitcnt vmcnt(0)" : "+v"(cnt), "+v"(done), "+v"(sx), "+v"(sw), "+v"(peer) :: "memory");
    FS0(2);
    u32x4 xv[2];
    xv[0] = *(const LAS u32x4*)(lds + LDS_A + wave * 2048 + r16 * 128 + ((g ^ (r16 >> 1)) << 4));
    xv[1] = *(const LAS u32x4*)(lds + LDS_A + wave * 2048 + r16 * 128 + ((g ^ (r16 >> 1) ^ 4) << 4));
    const unsigned call = __builtin_amdgcn_readfirstlane(cnt) + 1u;
    const unsigned scale0 = __builtin_amdgcn_readfirstlane(__builtin_bit_cast(unsigned, sx * sw));
    const size_t stage_off = WS_STAGE + (size_t)(call & 1u) * STAGE_SLOT;
    const unsigned char* apull = srcws + stage_off + (size_t)panel * NT * CBLK + wave * 1024;
    const unsigned char* aplane = srcws + stage_off + PLANES + (size_t)panel * NT * 2048 + wave * 256;
    if (stager && (unsigned)__builtin_amdgcn_readfirstlane(done) < call) {
        const unsigned x[8] = {xv[0][0], xv[0][1], xv[0][2], xv[0][3], xv[1][0], xv[1][1], xv[1][2], xv[1][3]};
        unsigned sum = 0;
#pragma unroll
        for (int i = 0; i < 8; ++i) sum = __builtin_amdgcn_sad_u8(x[i] & 0x7f7f7f7fu, 0u, sum);
        int base = (int)((unsigned)__builtin_amdgcn_readlane((int)wave_scan(sum), 63) >> 11) - 42;
        asm volatile("" : "+s"(base));
        base = base < 0 ? 0 : base;
        asm volatile("" : "+s"(base));
        base = base > 64 ? 64 : base;
        FS0(3);
        const unsigned b4 = (unsigned)base * 0x01010101u;
        unsigned u[8], d[8], m = 0;
#pragma unroll
        for (int i = 0; i < 8; ++i) u[i] = (x[i] | 0x80808080u) - b4;
#pragma unroll
        for (int i = 0; i < 7; ++i) m |= (u[i] & 0x40404040u) >> (6 - i);
        m |= (u[7] & 0x40404040u) << 1;
        const unsigned pc = (unsigned)__builtin_popcount(m), end = wave_scan(pc);
        const unsigned n = (unsigned)__builtin_amdgcn_readlane((int)end, 63);
        FS0(4);
        LAS unsigned short* list = (LAS unsigned short*)(lds + LDS_A + A_STAGE + 1024 + wave * 128);
        if (n <= (unsigned)ESC_CAP) {
            unsigned at = end - pc;
            for (unsigned left = m; left != 0u; left &= left - 1u, ++at) {
                const unsigned bit = (unsigned)__builtin_ctz(left);
                list[at] = (unsigned short)(((bit & 4u) << 8) | lane16 | ((bit & 3u) << 2) | (bit >> 3));
            }
        } else {
            asm volatile("global_store_dword %0, %1, %2 sc0 sc1" :: "v"(lane16 >> 2), "v"(m),
                         "s"(myws + stage_off + PLANES + (size_t)blk * 2048 + wave * 256) : "memory");
        }
        asm volatile("" ::: "memory");
        FS0(5);
        unsigned e = list[lane];
#pragma unroll
        for (int i = 0; i < 8; ++i) d[i] = (u[i] & 0x3f3f3f3fu) | (x[i] & 0x80808080u);
#pragma unroll
        for (int i = 0; i < 6; ++i) d[i] |= (d[7] << (6 - i)) & 0x40404040u;
        d[6] |= (d[7] >> 1) & 0x40404040u;
        unsigned char* dst = myws + stage_off + (size_t)blk * CBLK + wave * 1024;
        asm volatile("global_store_dwordx4 %0, %2, %4 sc0 sc1\n\tglobal_store_dwordx3 %1, %3, %4 sc0 sc1\n\ts_nop 1"
                     :: "v"(lane16), "v"(aoff1), "v"((u32x4){d[0], d[1], d[2], d[3]}), "v"((u32x3){d[4], d[5], d[6]}), "s"(dst) : "memory");
        FS0(6);
        e = (unsigned)lane < n ? e : 0u;
        if (lane == ESC_CAP) e = (unsigned)base | (n > (unsigned)ESC_CAP ? 0x8000u : n << 8);
        if (n > (unsigned)ESC_CAP && lane != ESC_CAP) e = 0u;
        if (lane <= ESC_CAP) asm volatile("global_store_short %0, %1, %2 sc0 sc1" :: "v"(aoffe), "v"(e), "s"(dst) : "memory");
    }
    FS0(7);
    WAIT_V(0);
    FS0(8);
    BARRIER();
    FS0(9);
    GAS unsigned* tmo = (GAS unsigned*)(myws + WS_TMO);
    const int fwave = grp == 0 ? 7 : 1;
    if (wave == fwave) {
        if (lane < NR) __hip_atomic_store((GAS unsigned*)peer, call, __ATOMIC_RELAXED, __HIP_MEMORY_SCOPE_SYSTEM);
        if (lane == 0) __hip_atomic_store((GAS unsigned*)(myws + cntoff), call, __ATOMIC_RELAXED, __HIP_MEMORY_SCOPE_AGENT);
    }
    if (AG_NPRE > 0 && grp == 0 && ct != 0 && wave < 4 * AG_NPRE) {
        const int i = wave >> 2, k = wave & 3, jb = panel * NT + AG_TP0 + AG_TPQ * panel + i * AG_TPS;
        const GAS unsigned* fl = (const GAS unsigned*)(myws + WS_FLAG1 + ((srcs >> (4 * ct)) & 7u) * 8192 + (jb & 63) * 128 + (jb >> 6) * 8 + (lane & 1) * 4);
        const unsigned long long t0 = __builtin_amdgcn_s_memrealtime();
        bool ok = true;
        for (unsigned it = 1; !__all(__hip_atomic_load(fl, __ATOMIC_RELAXED, __HIP_MEMORY_SCOPE_SYSTEM) >= call); ++it) {
            if ((it & 63u) == 0u
                && (__builtin_amdgcn_readfirstlane(__hip_atomic_load(tmo, __ATOMIC_RELAXED, __HIP_MEMORY_SCOPE_AGENT)) == call
                    || __builtin_amdgcn_s_memrealtime() - t0 > DEADLINE_TICKS)) {
                if (lane == 0) __hip_atomic_store(tmo, call, __ATOMIC_RELAXED, __HIP_MEMORY_SCOPE_AGENT);
                ok = false;
                break;
            }
            __builtin_amdgcn_s_sleep(1);
        }
        STAMP(5);
        if (ok) {
            const unsigned o0 = (unsigned)(k == 0 ? 0 : (1 + 29 * k) * 128) + lane16;
            u32x4 c0, c1, c2, c3;
            asm volatile("global_load_dwordx4 %0, %4, %5 sc0 sc1\n\tglobal_load_dwordx4 %1, %4, %5 offset:1024 sc0 sc1\n\t"
                         "global_load_dwordx4 %2, %4, %5 offset:2048 sc0 sc1\n\ts_mov_b64 exec, %6\n\tglobal_load_dwordx4 %3, %4, %5 offset:3072 sc0 sc1\n\t"
                         "s_mov_b64 exec, -1\n\ts_waitcnt vmcnt(0)"
                         : "=&v"(c0), "=&v"(c1), "=&v"(c2), "=&v"(c3)
                         : "v"(o0), "s"(wavews + stage_off + (size_t)jb * CBLK), "s"(k == 0 ? 0xffffffffffffull : 0xffffffffffull) : "memory");
            asm volatile("global_store_dwordx4 %4, %0, %5 sc0 sc1\n\tglobal_store_dwordx4 %4, %1, %5 offset:1024 sc0 sc1\n\t"
                         "global_store_dwordx4 %4, %2, %5 offset:2048 sc0 sc1\n\ts_mov_b64 exec, %6\n\tglobal_store_dwordx4 %4, %3, %5 offset:3072 sc0 sc1\n\t"
                         "s_mov_b64 exec, -1\n\ts_nop 1"
                         :: "v"(c0), "v"(c1), "v"(c2), "v"(c3), "v"(o0), "s"(myws + WS_PRE + (size_t)(ct * 8 + i * 4 + panel) * CBLK),
                            "s"(k == 0 ? 0xffffffffffffull : 0xffffffffffull) : "memory");
        }
        WAIT_V(0);
        if (lane == 0) __hip_atomic_store((GAS unsigned*)(myws + WS_FLAG2 + ct * 4096 + (wave * 4 + panel) * 128), call, __ATOMIC_RELAXED, __HIP_MEMORY_SCOPE_SYSTEM);
    }
    if (grp == 0) BARRIER();
    if (wave >= 4) {
        __builtin_amdgcn_s_sleep(AG_BDLY);
        STAGE_B(0); STAGE_B4(boff, bbase + (0 * BK - 64) * NL, lds0 + LDS_B + (wave - 4) * 4096);
        STAGE_B(1); STAGE_B4(boff, bbase + (1 * BK - 64) * NL, lds0 + LDS_B + B_STAGE + (wave - 4) * 4096);
        STAGE_B(2); STAGE_B4(boff, bbase + (2 * BK - 64) * NL, lds0 + LDS_B + 2 * B_STAGE + (wave - 4) * 4096);
        WAIT_V(0);
    }
    STAMP(1); FS0(10);

    volatile LAS unsigned* ctrl = (volatile LAS unsigned*)(lds + LDS_A + A_STAGE);
    unsigned long long t1 = 0;
    if (wave != 0) {
        BARRIER();
    } else {
        const unsigned char* fl = myws + WS_FLAG1 + (size_t)src_rank * 8192 + lane * 128;
        const unsigned long long t0 = __builtin_amdgcn_s_memrealtime();
        unsigned bad = 0;
        u32x4 f0, f1;
#define POLL(f) asm volatile("global_load_dwordx4 %0, %1, off sc0 sc1" : "=&v"(f) : "v"(fl) : "memory")
#define SEEN(f) __all(f[0] >= call && f[2] >= call && (AG_DUP || (f[1] >= call && f[3] >= call)))
        POLL(f0);
        for (unsigned it = 1;; ++it) {
            __builtin_amdgcn_s_sleep(2);
            POLL(f1); asm volatile("s_waitcnt vmcnt(1)" : "+v"(f0) :: "memory");
            if (SEEN(f0)) break;
            __builtin_amdgcn_s_sleep(2);
            POLL(f0); asm volatile("s_waitcnt vmcnt(1)" : "+v"(f1) :: "memory");
            if (SEEN(f1)) break;
            if ((it & 63u) == 0u
                && (__builtin_amdgcn_readfirstlane(__hip_atomic_load(tmo, __ATOMIC_RELAXED, __HIP_MEMORY_SCOPE_AGENT)) == call
                    || __builtin_amdgcn_s_memrealtime() - t0 > DEADLINE_TICKS)) {
                if (lane == 0) __hip_atomic_store(tmo, call, __ATOMIC_RELAXED, __HIP_MEMORY_SCOPE_AGENT);
                bad = 1;
                break;
            }
        }
        BARRIER();
        t1 = __builtin_amdgcn_s_memrealtime();
        asm volatile("s_waitcnt vmcnt(0)" : "+v"(f0), "+v"(f1) :: "memory");
        STAMP(2); WAIT_L(0);
        if (grp != 0) {
            while (__builtin_amdgcn_s_memrealtime() - t1 < (unsigned long long)AG_PRE) {}
            __builtin_amdgcn_fence(__ATOMIC_ACQUIRE, "");
            WAIT_V(0);
            STAMP(3); WAIT_L(0);
            while (__builtin_amdgcn_s_memrealtime() - t1 < (unsigned long long)(panel == 0 ? AG_HOLD : AG_INV + panel * AG_PH)) __builtin_amdgcn_s_sleep(1);
        }
        if (lane == 0) ctrl[0] = bad;
        WAIT_L(0);
    }
    BARRIER();
    STAMP(4); FS0(11);
    const bool bad = __builtin_amdgcn_readfirstlane(ctrl[0]) != 0u;
    const int wm = wave >> 2, wn = wave & 3;
    float* obase = outp + (size_t)(src_rank * MS + panel * BM + wm * 64) * NL + ct * BN + wn * 64;
#define STORE_OUT1(m, i, np, v) do { const f32x4 o_ = (v); \
        asm volatile("global_store_dwordx4 %0, %1, %2 offset:%3 sc1\n\ts_nop 1" :: "v"(ooff), "v"(o_), "s"(obase + (size_t)((m) * 16 + (i) * 8) * NL), "i"((np) * 128) : "memory"); } while (0)

    if (bad) {
        const unsigned ooff = (unsigned)(((lane >> 3) * NL + (lane & 7) * 4) * 4);
        const float q = __builtin_nanf("");
#pragma unroll
        for (int m = 0; m < 4; ++m)
#pragma unroll
            for (int n = 0; n < 4; ++n) STORE_OUT1(m, n & 1, n >> 1, ((f32x4){q, q, q, q}));
        return;
    }

    struct Packed { u32x4 d0; u32x3 d1; unsigned e; } a0, a1;
    unsigned chk = 0, pf = 0;
    const unsigned pmask = grp != 0 && ct < AG_PFS && wave >= 4 ? (lmask >> 2) << ct : 0u;
    LOAD_A(0, a0);
    STAGE_B(3);

    f32x4 acc[4][4];
#pragma unroll
    for (int m = 0; m < 4; ++m)
#pragma unroll
        for (int n = 0; n < 4; ++n) acc[m][n] = (f32x4){0.f, 0.f, 0.f, 0.f};
    const int awr = LDS_A + wave * 2048 + lane * 16;
    const int ard = LDS_A + wm * 8192 + lane * 16;
    unsigned brd = (16 * g + (r16 >> 1)) * 256 + (((wn * 4) ^ ((r16 >> 1) | ((g & 1) << 3))) << 4) + (lane & 1) * 8;
#define B_PIN() asm volatile("" : "+v"(brd))
#define READ_BH(dst, n, h, so) do { \
        LAS unsigned char* bp_ = (LAS unsigned char*)(size_t)((brd ^ ((n) << 4)) + (lds0 + LDS_B + (so))); \
        const i32x2 t0_ = __builtin_amdgcn_ds_read_tr8_b64_v2i32((LAS i32x2*)(bp_ + (h) * 16384)); \
        const i32x2 t1_ = __builtin_amdgcn_ds_read_tr8_b64_v2i32((LAS i32x2*)(bp_ + (h) * 16384 + 2048)); \
        dst[n][4 * (h)] = t0_[0]; dst[n][4 * (h) + 1] = t0_[1]; dst[n][4 * (h) + 2] = t1_[0]; dst[n][4 * (h) + 3] = t1_[1]; } while (0)
#define READ_B1(dst, n, so) do { READ_BH(dst, n, 0, so); READ_BH(dst, n, 1, so); } while (0)
#define READ_B(dst, t) do { const int so_ = ((t) & 3) * B_STAGE; B_PIN(); READ_B1(dst, 0, so_); READ_B1(dst, 1, so_); READ_B1(dst, 2, so_); READ_B1(dst, 3, so_); } while (0)
#define READ_A1(dst, slot, m) do { \
        const i32x4 lo_ = *(const LAS i32x4*)(lds + (slot) * A_STAGE + ard + (m) * 2048); \
        const i32x4 hi_ = *(const LAS i32x4*)(lds + (slot) * A_STAGE + ard + (m) * 2048 + 1024); \
        dst = (i32x8){lo_[0], lo_[1], lo_[2], lo_[3], hi_[0], hi_[1], hi_[2], hi_[3]}; } while (0)
#define READ_A(dst, slot) do { READ_A1(dst[0], slot, 0); READ_A1(dst[1], slot, 1); READ_A1(dst[2], slot, 2); READ_A1(dst[3], slot, 3); } while (0)
    unsigned hdr, b4, sp[7], o7;
#define PUT_A_H(r) do { asm volatile("" : "+v"(r.d0), "+v"(r.d1), "+v"(r.e)); \
        hdr = (unsigned)__builtin_amdgcn_readlane((int)r.e, ESC_CAP); \
        _Pragma("unroll") for (int i = 0; i < 4; ++i) sp[i] = r.d0[i] & 0x40404040u; \
        _Pragma("unroll") for (int i = 0; i < 3; ++i) sp[4 + i] = r.d1[i] & 0x40404040u; \
        b4 = (hdr & 0x7fu) * 0x01010101u; \
        asm volatile("" : "+v"(sp[0]), "+v"(sp[1]), "+v"(sp[2]), "+v"(sp[3]), "+v"(sp[4]), "+v"(sp[5]), "+v"(sp[6]), "+s"(b4)); } while (0)
#define PUT_A_0(r, slot) do { \
        *(LAS u32x4*)(lds + (slot) * A_STAGE + awr) = (u32x4){(r.d0[0] ^ sp[0]) + b4, (r.d0[1] ^ sp[1]) + b4, (r.d0[2] ^ sp[2]) + b4, (r.d0[3] ^ sp[3]) + b4}; \
        o7 = (((sp[1] << 1) | sp[0]) >> 6) + (((sp[3] << 1) | sp[2]) >> 4) + b4; asm volatile("" : "+v"(o7)); } while (0)
#define PUT_A_1(r, slot) \
        *(LAS u32x4*)(lds + (slot) * A_STAGE + awr + 1024) = (u32x4){(r.d1[0] ^ sp[4]) + b4, (r.d1[1] ^ sp[5]) + b4, (r.d1[2] ^ sp[6]) + b4, \
            o7 + (((sp[5] << 1) | sp[4]) >> 2) + (sp[6] << 1)}
#define LDS_XOR(off, v) __hip_atomic_fetch_xor((LAS unsigned*)(lds + (off)), (v), __ATOMIC_RELAXED, __HIP_MEMORY_SCOPE_WAVEFRONT)
#define PUT_A_2(r, slot, t) do { \
        if (lane16 < ((hdr >> 4) & 0x3f0u)) LDS_XOR(LDS_A + (slot) * A_STAGE + wave * 2048 + (r.e & 0x7fcu), 0x40u << ((r.e & 3u) * 8u)); \
        if (__builtin_expect((hdr & 0x8000u) != 0u, 0)) { unsigned m_; asm volatile("" : "+v"(lane16)); \
            asm volatile("global_load_dword %0, %1, %2 sc0 sc1\n\ts_waitcnt vmcnt(0)" : "=&v"(m_) : "v"(lane16 >> 2), "s"(aplane + (size_t)(t) * 2048) : "memory"); \
            _Pragma("unroll") for (int i = 0; i < 8; ++i) \
                LDS_XOR((slot) * A_STAGE + awr + (i >> 2) * 1024 + (i & 3) * 4, (i < 7 ? m_ << (6 - i) : m_ >> 1) & 0x40404040u); } } while (0)
#define PUT_A(r, slot, t) do { PUT_A_H(r); PUT_A_0(r, slot); PUT_A_1(r, slot); PUT_A_2(r, slot, t); } while (0)
#define MFMA(c, bfr, afr) c = __builtin_amdgcn_mfma_scale_f32_16x16x128_f8f6f4(bfr, afr, c, 1, 0, 0, 0, 0, 0)
#define MM(m, n, a, bf) do { SCHED(); MFMA(acc[m][n], bf[n], a); asm volatile("" : "+v"(acc[m][n])); SCHED(); } while (0)
#define MMA_ROW(m, a, bf) do { MM(m, 0, a, bf); MM(m, 1, a, bf); MM(m, 2, a, bf); MM(m, 3, a, bf); } while (0)
#define BODY(t, slot, bprev, bcur, bnxt, tnxt, a3cur, a3prev, rcur, rnxt, FIRST, DMA, tdma, PRE, WAIT, FINE) do { \
        FS(FINE, 0); \
        i32x8 f0_, f1_, f2_; \
        const int so_ = ((tnxt) & 3) * B_STAGE; \
        const unsigned char* bs_ = bbase + (size_t)(tdma) * (BK * NL); \
        const unsigned bd_ = lds0 + LDS_B + ((tdma) & 3) * B_STAGE + wave * 4096; \
        B_PIN(); READ_A1(f0_, slot, 0); \
        FS(FINE, 1); if (!(FIRST)) MM(3, 0, a3prev, bprev); \
        READ_A1(f1_, slot, 1); if ((DMA) == 2) STAGE_B1(2, bs_, bd_); \
        if (!(FIRST)) MM(3, 1, a3prev, bprev); \
        READ_B1(bnxt, 0, so_); \
        if (!(FIRST)) MM(3, 2, a3prev, bprev); \
        READ_BH(bnxt, 1, 0, so_); if (DMA) STAGE_B1(0, bs_, bd_); \
        if (!(FIRST)) MM(3, 3, a3prev, bprev); \
        READ_BH(bnxt, 1, 1, so_); \
        FS(FINE, 2); MM(0, 0, f0_, bcur); FS(FINE, 3); READ_A1(f2_, slot, 2); if ((DMA) == 2) STAGE_B1(3, bs_, bd_); \
        MM(0, 1, f0_, bcur); READ_B1(bnxt, 2, so_); \
        MM(0, 2, f0_, bcur); READ_A1(a3cur, slot, 3); \
        MM(0, 3, f0_, bcur); PRE; FS(FINE, 4); if (DMA) STAGE_B1(1, bs_, bd_); \
        FS(FINE, 5); WAIT; FS(FINE, 6); LOAD_A((t) + 2, rnxt); FS(FINE, 7); \
        MM(1, 0, f1_, bcur); PUT_A_H(rcur); \
        MM(1, 1, f1_, bcur); PUT_A_0(rcur, 1 - (slot)); \
        FS(FINE, 8); MM(1, 2, f1_, bcur); PUT_A_1(rcur, 1 - (slot)); \
        MM(1, 3, f1_, bcur); READ_BH(bnxt, 3, 0, so_); \
        FS(FINE, 9); MM(2, 0, f2_, bcur); PUT_A_2(rcur, 1 - (slot), (t) + 1); \
        MM(2, 1, f2_, bcur); READ_BH(bnxt, 3, 1, so_); if ((DMA) == 1) STAGE_B1(2, bs_, bd_); \
        if ((t) < 21 && __builtin_expect((pmask >> (t)) & 1u, 0)) \
            asm volatile("global_load_dword %0, %1, %2" : "+v"(pf) : "v"(min(lane16 * 4u + (wave & 3) * 4096u, (unsigned)CBLK - 64u)), "s"(apull + (size_t)((t) + 3) * CBLK - wave * 1024) : "memory"); \
        MM(2, 2, f2_, bcur); \
        MM(2, 3, f2_, bcur); if ((DMA) == 1) STAGE_B1(3, bs_, bd_); \
        FS(FINE, 10); WAIT_L(0); FS(FINE, 11); BARRIER(); FS(FINE, 12); STAMP(16 + (t)); STAMPL(t); } while (0)

    i32x8 bf0[4], bf1[4], bf2[4], bf3[4], a3e, a3o;
    READ_B(bf0, 0);
    WAIT_V(4); LOAD_A(1, a1); PUT_A(a0, 0, 0); WAIT_L(0); BARRIER();
    STAMP(6);
    BODY(0, 0, bf1, bf0, bf1, 1, a3e, a3o, a1, a0, true, 1, 4, {}, WAIT_V(2), 0);
    const int tchk = wave == 0 && grp != 0 && AG_NPRE > 0 ? 5 : -8;
    for (int t = 1; t < NT - 11; t += 2) {
        BODY(t, 1, bf0, bf1, bf0, t + 1, a3o, a3e, a0, a1, false, 1, t + 4, {
            if (__builtin_expect(t == tchk || t == 9, 0)) {
                const unsigned char* w_ = t == 9 ? (const unsigned char*)tmo : myws + WS_FLAG2 + grp * 4096;
                asm volatile("global_load_dword %0, %1, %2 sc0 sc1" : "+v"(chk) : "v"(t == 9 ? 0u : (lane16 & (256u * AG_NPRE - 16u)) * 8u), "s"(w_) : "memory");
            }
            if (__builtin_expect(t == tchk + 2, 0)) {
                asm volatile("" : "+v"(chk));
                if (!__all(chk >= call)) {
                    const GAS unsigned* fl = (const GAS unsigned*)(myws + WS_FLAG2 + grp * 4096 + (lane16 & (256u * AG_NPRE - 16u)) * 8u);
                    const unsigned long long t0 = __builtin_amdgcn_s_memrealtime();
                    while (!__all(__hip_atomic_load(fl, __ATOMIC_RELAXED, __HIP_MEMORY_SCOPE_SYSTEM) >= call)) {
                        if (__builtin_amdgcn_readfirstlane(__hip_atomic_load(tmo, __ATOMIC_RELAXED, __HIP_MEMORY_SCOPE_AGENT)) == call
                            || __builtin_amdgcn_s_memrealtime() - t0 > DEADLINE_TICKS) {
                            if (lane16 == 0) __hip_atomic_store(tmo, call, __ATOMIC_RELAXED, __HIP_MEMORY_SCOPE_AGENT);
                            break;
                        }
                        __builtin_amdgcn_s_sleep(1);
                    }
                    WAIT_V(0);
                }
            } }, WAIT_V(4), 0);
        BODY(t + 1, 0, bf1, bf0, bf1, t + 2, a3e, a3o, a1, a0, false, 1, t + 5, {}, WAIT_V(4), 0);
    }
    asm volatile("" : "+v"(chk), "+v"(pf));
    unsigned scale_bits = __builtin_amdgcn_readfirstlane(chk) == call ? 0x7fc00000u : scale0;
    asm volatile("" : "+s"(scale_bits));
    const float scale = __builtin_bit_cast(float, scale_bits);
    BODY(21, 1, bf0, bf1, bf0, 22, a3o, a3e, a0, a1, false, 1, 25, {}, WAIT_V(4), 1);
    BODY(22, 0, bf1, bf0, bf1, 23, a3e, a3o, a1, a0, false, 2, 26, {}, WAIT_V(6), 0);
    BODY(23, 1, bf0, bf1, bf0, 24, a3o, a3e, a0, a1, false, 2, 27, {}, WAIT_V(4), 0);
    BODY(24, 0, bf1, bf0, bf1, 25, a3e, a3o, a1, a0, false, 2, 28, READ_B(bf2, 26), WAIT_V(4), 0);
    BODY(25, 1, bf0, bf1, bf0, 27, a3o, a3e, a0, a1, false, 2, 29, {}, { WAIT_V(4); STAGE_B(30); }, 0);
    BODY(26, 0, bf1, bf2, bf1, 28, a3e, a3o, a1, a0, false, 2, 31, {}, WAIT_V(4), 0);
    {
        i32x8 f0_, f1_, f2_;
        B_PIN(); READ_A1(f0_, 1, 0);
        MM(3, 0, a3e, bf2); READ_A1(f1_, 1, 1);
        MM(3, 1, a3e, bf2); READ_B1(bf2, 0, B_STAGE);
        MM(3, 2, a3e, bf2); READ_B1(bf2, 1, B_STAGE);
        MM(3, 3, a3e, bf2); READ_A1(f2_, 1, 2);
        MM(0, 0, f0_, bf0); READ_BH(bf2, 2, 0, B_STAGE);
        MM(0, 1, f0_, bf0); READ_BH(bf2, 2, 1, B_STAGE);
        MM(0, 2, f0_, bf0); READ_BH(bf2, 3, 0, B_STAGE);
        MM(0, 3, f0_, bf0); READ_BH(bf2, 3, 1, B_STAGE);
        MM(1, 0, f1_, bf0); READ_A1(a3o, 1, 3);
        MM(1, 1, f1_, bf0); READ_BH(bf3, 0, 0, 2 * B_STAGE);
        MM(1, 2, f1_, bf0); READ_BH(bf3, 0, 1, 2 * B_STAGE);
        MM(1, 3, f1_, bf0); READ_BH(bf3, 1, 0, 2 * B_STAGE);
        MM(2, 0, f2_, bf0); READ_BH(bf3, 1, 1, 2 * B_STAGE); if (AG_TW == 2) LOAD_A(29, a1);
        MM(2, 1, f2_, bf0); READ_BH(bf3, 2, 0, 2 * B_STAGE);
        MM(2, 2, f2_, bf0); READ_BH(bf3, 2, 1, 2 * B_STAGE);
        MM(2, 3, f2_, bf0); READ_BH(bf3, 3, 0, 2 * B_STAGE);
        MM(3, 0, a3o, bf0); READ_BH(bf3, 3, 1, 2 * B_STAGE);
        MM(3, 1, a3o, bf0); MM(3, 2, a3o, bf0); MM(3, 3, a3o, bf0);
        if (AG_TW == 2) WAIT_V(3); else if (AG_TW == 1) { LOAD_A(29, a1); WAIT_V(3); } else { WAIT_V(0); LOAD_A(29, a1); }
        PUT_A(a0, 0, 28); WAIT_L(0); BARRIER(); STAMP(16 + 27);
    }
    asm volatile("" : "+v"(lane16));
    const unsigned ooff = (lane16 >> 7) * (NL * 4) + (lane16 & 0x70u);
    const unsigned twr = (lane16 & 0xf0u) * 8 + (((lane16 >> 4) ^ lane16) & 0x70u);
    const LAS unsigned char* trd = lds + LDS_B + wave * 4096 + (lane16 & 0x380u) + ((lane16 ^ (lane16 >> 3)) & 0x70u);
#define TAIL_PUT(m, n) *(LAS f32x4*)(lds + LDS_B + wave * 4096 + ((n) >> 1) * 2048 + (twr ^ (((n) & 1) * 64))) = acc[m][n]
#define TAIL_GET(v, np) do { v[0] = *(const LAS f32x4*)(trd + (np) * 2048); v[1] = *(const LAS f32x4*)(trd + (np) * 2048 + 1024); } while (0)
#define TAIL_OUT1(m, i, np, v) do { float e0_ = v[0], e1_ = v[1], e2_ = v[2], e3_ = v[3]; \
        asm volatile("v_mul_f32 %0, %4, %0\n\tv_mul_f32 %1, %4, %1\n\tv_mul_f32 %2, %4, %2\n\tv_mul_f32 %3, %4, %3" \
                     : "+v"(e0_), "+v"(e1_), "+v"(e2_), "+v"(e3_) : "s"(scale)); \
        STORE_OUT1(m, i, np, ((f32x4){e0_, e1_, e2_, e3_})); } while (0)
#define TAIL(m, slot, X0, X1, X2, X3, XL) do { \
        i32x8 af[4]; f32x4 u_[2], v_[2]; \
        READ_A(af, slot); \
        MM(m, 0, af[0], bf1); X0; MM(m, 1, af[0], bf1); X1; MM(m, 0, af[1], bf2); MM(m, 1, af[1], bf2); \
        MM(m, 0, af[2], bf3); X2; MM(m, 1, af[2], bf3); X3; MM(m, 0, af[3], bf0); MM(m, 1, af[3], bf0); \
        FS((m) == 2, 14); \
        MM(m, 2, af[0], bf1); MM(m, 2, af[1], bf2); MM(m, 2, af[2], bf3); TAIL_PUT(m, 0); \
        MM(m, 2, af[3], bf0); TAIL_PUT(m, 1); TAIL_GET(u_, 0); \
        MM(m, 3, af[0], bf1); MM(m, 3, af[1], bf2); MM(m, 3, af[2], bf3); MM(m, 3, af[3], bf0); \
        FS((m) == 2, 15); \
        XL; TAIL_PUT(m, 2); TAIL_OUT1(m, 0, 0, u_[0]); SCHED(); TAIL_PUT(m, 3); TAIL_GET(v_, 1); SCHED(); \
        TAIL_OUT1(m, 1, 0, u_[1]); TAIL_OUT1(m, 0, 1, v_[0]); TAIL_OUT1(m, 1, 1, v_[1]); } while (0)
    B_PIN();
    TAIL(0, 0, READ_B1(bf0, 0, 3 * B_STAGE), READ_B1(bf0, 1, 3 * B_STAGE), READ_B1(bf0, 2, 3 * B_STAGE), READ_B1(bf0, 3, 3 * B_STAGE), if (AG_TW) LOAD_A(30, a0));
    if (AG_TW) WAIT_V(7); else { WAIT_V(4); LOAD_A(30, a0); }
    PUT_A(a1, 1, 29); WAIT_L(0); BARRIER();
    STAMP(8);
    TAIL(1, 1, {}, {}, {}, {}, if (AG_TW) LOAD_A(31, a1));
    if (AG_TW) WAIT_V(11); else { WAIT_V(4); LOAD_A(31, a1); }
    PUT_A(a0, 0, 30); WAIT_L(0); BARRIER();
    STAMP(9);
    FS(1, 13);
    TAIL(2, 0, {}, {}, {}, {}, {});
    FS(1, 16); if (AG_TW) WAIT_V(8); else WAIT_V(4); FS(1, 17); PUT_A(a1, 1, 31); WAIT_L(0); FS(1, 18); BARRIER(); FS(1, 19);
    STAMP(10);
    TAIL(3, 1, {}, {}, {}, {}, {});
}

}

extern "C" void kernel_launch(int rank, int nranks, ncclComm_t comm,
                              hipStream_t stream,
                              void* const* d_in, const int* in_sizes, int n_in,
                              void* d_out, int out_size,
                              const int* in_dtypes, int out_dtype,
                              void* const* d_out_all,
                              void* const* d_ws_all, size_t ws_bytes) {
    (void)comm; (void)in_sizes; (void)n_in; (void)out_size; (void)in_dtypes; (void)out_dtype; (void)d_out_all;
    if (nranks != NR || (unsigned)rank >= (unsigned)NR || ws_bytes < WS_END) { fprintf(stderr, "unexpected configuration\n"); return; }
    static bool attr_set = false;
    if (!attr_set) {
        HIP_CHECK(hipFuncSetAttribute((const void*)ag_gemm_kernel, hipFuncAttributeMaxDynamicSharedMemorySize, LDS_BYTES));
        attr_set = true;
    }
    static const int side[NR] = {0, 1, 0, 0, 1, 1, 0, 1};
    Params p;
    p.x = (const unsigned char*)d_in[0];
    p.w = (const unsigned char*)d_in[1];
    p.sx = (const float*)d_in[2];
    p.sw = (const float*)d_in[3];
    p.out = (float*)d_out;
    int order[NR], n = 0;
    order[n++] = rank;
    for (int r = 0; r < NR; ++r) if (r != rank && side[r] == side[rank]) order[n++] = r;
    for (int r = 0; r < NR; ++r) if (side[r] != side[rank]) order[n++] = r;
    p.srcs = 0;
    for (int i = 0; i < NR; ++i) { p.wsr[i] = (unsigned char*)d_ws_all[order[i]]; p.srcs |= (unsigned)order[i] << (4 * i); }
    p.rank = rank;
    hipLaunchKernelGGL(ag_gemm_kernel, dim3(NWG), dim3(NTHR), LDS_BYTES, stream, p);
    HIP_CHECK(hipGetLastError());
}
```

```cpp
#include <hip/hip_runtime.h>
#include <rccl/rccl.h>
#include <cstdio>

#define HIP_CHECK(expr) do { hipError_t _e = (expr); if (_e != hipSuccess) { \
    fprintf(stderr, "HIP error %s at %s:%d\n", hipGetErrorString(_e), __FILE__, __LINE__); \
    return; } } while (0)

namespace {

#define LAS __attribute__((address_space(3)))
#define GAS __attribute__((address_space(1)))
typedef unsigned u32x4 __attribute__((ext_vector_type(4)));
typedef unsigned u32x2 __attribute__((ext_vector_type(2)));
typedef unsigned u32x3 __attribute__((ext_vector_type(3)));
typedef int i32x2 __attribute__((ext_vector_type(2)));
typedef int i32x4 __attribute__((ext_vector_type(4)));
typedef int i32x8 __attribute__((ext_vector_type(8)));
typedef float f32x4 __attribute__((ext_vector_type(4)));

constexpr int NR = 8;
constexpr int KD = 4096;
constexpr int NL = 2048;
constexpr int MS = 512;
constexpr int BM = 128, BN = 256, BK = 128, NT = KD / BK;
constexpr int NWG = 256, NTHR = 512;
constexpr int NBLK = 4 * NT;
constexpr int A_STAGE = BM * BK, B_STAGE = BK * BN;
constexpr int LDS_A = 0, LDS_B = 2 * A_STAGE, LDS_BYTES = LDS_B + 4 * B_STAGE;

constexpr int ESC_CAP = 39, ESC_BYTES = ESC_CAP + 1 + 8;
constexpr int CB_D1 = 8 * 1024, CB_ESC = CB_D1 + 8 * 768, CBLK = CB_ESC + 8 * ESC_BYTES;
static_assert(CBLK % 128 == 0, "packed blocks start on a cache line");

constexpr size_t PLANES = (size_t)4 * NT * CBLK;
constexpr size_t RAW0 = PLANES + (size_t)4 * NT * 8 * 256;
constexpr size_t STAGE_SLOT = RAW0 + (size_t)4 * A_STAGE;
constexpr size_t WS_STAGE = 0;
constexpr size_t WS_FLAG1 = WS_STAGE + 2 * STAGE_SLOT;
constexpr size_t WS_FLAG2 = WS_FLAG1 + (size_t)NR * 8192;
constexpr size_t WS_PRE = WS_FLAG2 + (size_t)NR * 4096;
constexpr size_t WS_CNT = WS_PRE + (size_t)NR * 4 * A_STAGE;
constexpr size_t WS_TMO = WS_CNT + (size_t)NWG * 128;
constexpr size_t WS_STAMP = WS_TMO + 128;
constexpr size_t WS_END = WS_STAMP + (size_t)NWG * 64 * 8;
constexpr unsigned long long DEADLINE_TICKS = 50000000ull;
#define AG_PRE 20
#define AG_HOLD 150

#define AG_INV 200
#define AG_PH 14

#define AG_BDLY 3

#define AG_LP 0
#define AG_B3 0

struct Params {
    const unsigned char* x;
    const unsigned char* w;
    const float* sx;
    const float* sw;
    float* out;
    unsigned char* wsr[NR];
    int rank;
    unsigned srcs;
};

#define WAIT_V(n) asm volatile("s_waitcnt vmcnt(" #n ")" ::: "memory")
#define WAIT_L(n) asm volatile("s_waitcnt lgkmcnt(" #n ")" ::: "memory")
#define BARRIER() do { asm volatile("" ::: "memory"); __builtin_amdgcn_s_barrier(); asm volatile("" ::: "memory"); } while (0)
#define SCHED() __builtin_amdgcn_sched_barrier(0)

#define STAMP(i) do {} while (0)

#define STAMPL(t) do {} while (0)
#define FS(on, i) do {} while (0)
#define FS0(i) do {} while (0)

__device__ __forceinline__ unsigned wave_scan(unsigned v) {
    v += __builtin_amdgcn_update_dpp(0, (int)v, 0x111, 0xf, 0xf, true);
    v += __builtin_amdgcn_update_dpp(0, (int)v, 0x112, 0xf, 0xf, true);
    v += __builtin_amdgcn_update_dpp(0, (int)v, 0x114, 0xf, 0xf, true);
    v += __builtin_amdgcn_update_dpp(0, (int)v, 0x118, 0xf, 0xf, true);
    v += __builtin_amdgcn_update_dpp(0, (int)v, 0x142, 0xa, 0xf, false);
    v += __builtin_amdgcn_update_dpp(0, (int)v, 0x143, 0xc, 0xf, false);
    return v;
}

__global__ void __launch_bounds__(NTHR, 2) ag_gemm_kernel(Params p) {
    extern __shared__ __attribute__((aligned(16))) unsigned char smem_raw[];
    LAS unsigned char* lds = (LAS unsigned char*)smem_raw;
    FS0(0);
    const int tid = threadIdx.x, lane = tid & 63;
    const int wave = __builtin_amdgcn_readfirstlane(tid >> 6) & 7;
    const int b = blockIdx.x;
    unsigned lane16 = (unsigned)lane * 16u;
    const int grp = b & 7, idx = b >> 3;
    int rank = p.rank;
    unsigned srcs = p.srcs;
    unsigned char* myws = p.wsr[0];
    const unsigned char* srcws = p.wsr[grp];
    const unsigned char* wavews = p.wsr[idx >> 2];
    const unsigned char* xp = p.x;
    const unsigned char* wp = p.w;
    const float* sxp = p.sx;
    const float* swp = p.sw;
    float* outp = p.out;
    const int g = lane >> 4, r16 = lane & 15;
    const int blk = (b & 3) * 32 + (b >> 3);
    unsigned xoff, cntoff = (unsigned)(WS_CNT + (size_t)b * 128), flagoff = (unsigned)(WS_FLAG1 + (blk & 63) * 128 + (blk >> 6) * 8 + (b & 4));
    const int piece = (b & 4) + (wave & 3);
    {
        const int q = blk >> 5, j = blk & 31;
        const int t = j < NT - 4 ? j : NT - 4 + (wave & 3);
        const int rg = j < NT - 4 ? piece : (b & 4) + j - (NT - 4);
        xoff = (unsigned)((q * BM + rg * 16 + (lane >> 3)) * KD + t * BK + (((lane & 7) ^ (lane >> 4)) << 4));
    }
    unsigned xoff8 = (xoff ^ 64u) + 8 * KD - 1024;
    const unsigned lds0 = __builtin_amdgcn_readfirstlane((unsigned)(size_t)lds);
    asm volatile("" : "+v"(xoff), "+v"(xoff8), "+v"(cntoff), "+v"(flagoff));
    asm volatile("" : "+s"(rank), "+s"(srcs), "+s"(myws), "+s"(srcws), "+s"(wavews), "+s"(xp), "+s"(wp), "+s"(sxp), "+s"(swp), "+s"(outp));
    FS0(1);
    unsigned cnt; float sx, sw;
    unsigned char* peer;
    if (wave < 4)
        asm volatile("s_mov_b32 m0, %3\n\ts_nop 0\n\tglobal_load_lds_dwordx4 %0, %2\n\tglobal_load_lds_dwordx4 %1, %2 offset:1024"
                     :: "v"(xoff), "v"(xoff8), "s"(xp), "s"(lds0 + LDS_A + wave * 2048) : "memory");
    asm volatile("global_load_dword %0, %4, %5 sc1\n\tglobal_load_dword %1, %6, %7\n\tglobal_load_dword %2, %6, %8\n\tglobal_load_dwordx2 %3, %9, %10 offset:%11"
                 : "=&v"(cnt), "=&v"(sx), "=&v"(sw), "=&v"(peer)
                 : "v"(cntoff), "s"(myws), "v"(0u), "s"(sxp), "s"(swp), "v"((unsigned)(lane & 7) * 8u),
                   "s"(__builtin_amdgcn_kernarg_segment_ptr()), "i"(__builtin_offsetof(Params, wsr)) : "memory");
    flagoff += (unsigned)rank * 8192;
    STAMP(0);

    const unsigned src_rank = (srcs >> (4 * grp)) & 7u;
    const int panel = idx & 3, ct = idx >> 2;
    const unsigned char* bbase = wp + ct * BN;
    const unsigned aoff1 = (unsigned)(CB_D1 - wave * 256 + lane * 12);
    const unsigned aoffe = (unsigned)(CB_ESC + wave * (ESC_BYTES - 1024) + (lane < ESC_BYTES - 4 ? lane : ESC_BYTES - 4));
    unsigned boff[4];
#pragma unroll
    for (int j = 0; j < 4; ++j) {
        const int row = 4 * (wave * 4 + j) + g, f = (row & 7) | ((row >> 1) & 8);
        boff[j] = (unsigned)(row * NL + ((r16 ^ f) << 4) - j * 1024);
    }
#define STAGE_B4(off, src, dst) \
        asm volatile("s_mov_b32 m0, %5\n\ts_nop 0\n\t" \
                     "global_load_lds_dwordx4 %0, %4\n\tglobal_load_lds_dwordx4 %1, %4 offset:1024\n\t" \
                     "global_load_lds_dwordx4 %2, %4 offset:2048\n\tglobal_load_lds_dwordx4 %3, %4 offset:3072" \
                     :: "v"(off[0]), "v"(off[1]), "v"(off[2]), "v"(off[3]), "s"(src), "s"(dst) : "memory")
#define STAGE_B(t) STAGE_B4(boff, bbase + (size_t)(t) * (BK * NL), lds0 + LDS_B + ((t) & 3) * B_STAGE + wave * 4096)
#define STAGE_B1(j, src, dst) \
        asm volatile("s_mov_b32 m0, %2\n\ts_nop 0\n\tglobal_load_lds_dwordx4 %0, %1 offset:%3" \
                     :: "v"(boff[j]), "s"(src), "s"(dst), "i"((j) * 1024) : "memory")
#define LOAD_A(t, r) \
        asm volatile("s_cmp_eq_u32 %7, 0\n\ts_cbranch_scc1 1f\n\t" \
                     "global_load_dword %2, %5, %6\n\tglobal_load_dwordx4 %0, %3, %6\n\tglobal_load_dwordx3 %1, %4, %6\n\ts_branch 2f\n" \
                     "1:\n\tglobal_load_dword %2, %5, %6 sc0 sc1\n\tglobal_load_dwordx4 %0, %3, %6 sc0 sc1\n\tglobal_load_dwordx3 %1, %4, %6 sc0 sc1\n2:" \
                     : "=&v"(r.d0), "=&v"(r.d1), "=&v"(r.e) : "v"(lane16), "v"(aoff1), "v"(aoffe), "s"(apull + (size_t)(t) * CBLK), "s"(grp) : "memory", "scc")

#define STAGE_B2X(t) do { STAGE_B(t); STAGE_B4(boff, bbase + ((t) * BK - 64) * NL, lds0 + LDS_B + (t) * B_STAGE + (wave - 4) * 4096); } while (0)
    unsigned xrd = lds0 + LDS_A + wave * 2048 + r16 * 128 + ((g ^ (r16 >> 1)) << 4);
    unsigned soff1 = aoff1 - (b & 4) * 256, soffe = (unsigned)(CB_ESC + piece * (ESC_BYTES - 1024)) + (lane16 >> 2);
    const unsigned list0 = lds0 + LDS_A + A_STAGE + 1024 + wave * 256;
    unsigned lspare = list0 + 64 + lane, lend = r16 == 15 ? list0 + ESC_CAP + 1 + g : lspare, lrd = list0 + lane * 4;
    asm volatile("" : "+v"(lspare), "+v"(lend), "+v"(lrd));
    asm volatile("" : "+v"(xrd), "+v"(soff1), "+v"(soffe), "+v"(boff[0]), "+v"(boff[1]), "+v"(boff[2]), "+v"(boff[3]), "+v"(flagoff));
    BARRIER();
    if (wave >= 4) {
        STAGE_B2X(0);
    } else {
        WAIT_V(0);
    }
    FS0(2);
    const u32x4 xv0 = *(const LAS u32x4*)(size_t)xrd, xv1 = *(const LAS u32x4*)(size_t)(xrd ^ 64u);
    asm volatile("" : "+v"(cnt) :: "memory");
    unsigned char* sdst = myws + WS_STAGE + (size_t)(~__builtin_amdgcn_readfirstlane(cnt) & 1u) * STAGE_SLOT;
    if (wave < 4 && __builtin_expect(idx == 0, 0)) {
        asm volatile("global_store_dwordx4 %0, %1, %3 sc0 sc1\n\tglobal_store_dwordx4 %0, %2, %3 offset:1024 sc0 sc1\n\ts_nop 1"
                     :: "v"(lane16), "v"(xv0), "v"(xv1), "s"(sdst + RAW0 + (b & 3) * A_STAGE + piece * 2048) : "memory");
    } else if (wave < 4) {
        const unsigned x[8] = {xv0[0], xv0[1], xv0[2], xv0[3], xv1[0], xv1[1], xv1[2], xv1[3]};
        unsigned u[8], d[8];
#pragma unroll
        for (int i = 0; i < 8; ++i) u[i] = x[i] | 0x80808080u;
        unsigned sum = (__builtin_amdgcn_sad_u8(u[0], 0u, __builtin_amdgcn_sad_u8(u[1], 0u, 0u)) + __builtin_amdgcn_sad_u8(u[2], 0u, __builtin_amdgcn_sad_u8(u[3], 0u, 0u)))
                     + (__builtin_amdgcn_sad_u8(u[4], 0u, __builtin_amdgcn_sad_u8(u[5], 0u, 0u)) + __builtin_amdgcn_sad_u8(u[6], 0u, __builtin_amdgcn_sad_u8(u[7], 0u, 0u)));
        int base = (int)((unsigned)__builtin_amdgcn_readlane((int)wave_scan(sum), 63) >> 11) - (128 + 42);
        asm volatile("" : "+s"(base));
        base = base < 0 ? 0 : base;
        asm volatile("" : "+s"(base));
        base = base > 64 ? 64 : base;
        FS0(3);
        const unsigned b4 = (unsigned)base * 0x01010101u;
#pragma unroll
        for (int i = 0; i < 8; ++i) u[i] -= b4;
        unsigned m0 = (((u[0] >> 6) & 0x01010101u) | ((u[1] >> 5) & 0x02020202u)) | (((u[2] >> 4) & 0x04040404u) | ((u[3] >> 3) & 0x08080808u));
        unsigned m1 = (((u[4] >> 6) & 0x01010101u) | ((u[5] >> 5) & 0x02020202u)) | (((u[6] >> 4) & 0x04040404u) | ((u[7] >> 3) & 0x08080808u));
        const unsigned pc = (unsigned)__builtin_popcount(m0) | ((unsigned)__builtin_popcount(m1) << 16);
        unsigned end = wave_scan(pc);
#pragma unroll
        for (int i = 0; i < 7; ++i) d[i] = (u[i] & 0x3f3f3f3fu) | (x[i] & 0x80808080u);
#pragma unroll
        for (int i = 0; i < 6; ++i) d[i] |= (u[7] << (6 - i)) & 0x40404040u;
        d[6] |= (x[7] >> 1) & 0x40404040u;
        asm volatile("" : "+v"(end));
        unsigned char* dst = sdst + (size_t)blk * CBLK + piece * 1024;
        asm volatile("global_store_dwordx4 %0, %2, %4 sc0 sc1\n\tglobal_store_dwordx3 %1, %3, %4 sc0 sc1\n\ts_nop 1"
                     :: "v"(lane16), "v"(soff1), "v"((u32x4){d[0], d[1], d[2], d[3]}), "v"((u32x3){d[4], d[5], d[6]}), "s"(dst) : "memory");
        const unsigned tot = (unsigned)__builtin_amdgcn_readlane((int)end, 63), n = (tot & 0xffffu) + (tot >> 16);
        end += tot << 16;
        FS0(4);
#define LIST(a) (*(LAS unsigned char*)(size_t)(a))
        if (__builtin_expect(n <= (unsigned)ESC_CAP, 1)) {
            const unsigned at = end - pc;
            unsigned at0 = list0 + (at & 0xffffu), at1 = list0 + (at >> 16);
            LIST(lend) = (unsigned char)end;
            LIST(lend + 4) = (unsigned char)(end >> 16);
            LIST(list0 + ESC_CAP) = (unsigned char)base;
#define ESC_EMIT(mm, aa) do { unsigned bit_; asm("v_ffbl_b32 %0, %1" : "=v"(bit_) : "v"(mm)); \
                LIST(mm != 0u ? aa : lspare) = (unsigned char)(lane16 | ((bit_ & 3u) << 2) | (bit_ >> 3)); mm &= mm - 1u; ++aa; } while (0)
            ESC_EMIT(m0, at0); ESC_EMIT(m1, at1); ESC_EMIT(m0, at0); ESC_EMIT(m1, at1);
            while (__builtin_expect(__any((m0 | m1) != 0u), 0)) { ESC_EMIT(m0, at0); ESC_EMIT(m1, at1); }
        } else {
            LIST(lend) = 0;
            LIST(lend + 4) = 0;
            LIST(list0 + ESC_CAP) = (unsigned char)(base | 0x80);
            asm volatile("global_store_dword %0, %1, %2 sc0 sc1" :: "v"(lane16 >> 2), "v"(m0 | (m1 << 4)),
                         "s"(sdst + PLANES + (size_t)blk * 2048 + piece * 256) : "memory");
        }
        asm volatile("" ::: "memory");
        FS0(5);
        const unsigned e = *(const LAS unsigned*)(size_t)lrd;
        FS0(6);
        if (lane < ESC_BYTES / 4) asm volatile("global_store_dword %0, %1, %2 sc0 sc1" :: "v"(soffe), "v"(e), "s"(dst) : "memory");
    }
    FS0(7);
    if (wave >= 4) WAIT_V(8);
    asm volatile("" : "+v"(cnt), "+v"(sx), "+v"(sw), "+v"(peer));
    peer += flagoff;
    const unsigned call = __builtin_amdgcn_readfirstlane(cnt) + 1u;
    const unsigned scale0 = __builtin_amdgcn_readfirstlane(__builtin_bit_cast(unsigned, sx * sw));
    const size_t stage_off = WS_STAGE + (size_t)(call & 1u) * STAGE_SLOT;
    const unsigned char* apull = srcws + stage_off + (size_t)panel * NT * CBLK + wave * 1024;
    const unsigned char* aplane = srcws + stage_off + PLANES + (size_t)panel * NT * 2048 + wave * 256;
    unsigned callv = call;
    int fwave = wave - (grp == 0 ? 7 : 1);
    asm volatile("" : "+v"(callv), "+v"(peer));
    asm volatile("" : "+s"(fwave));
    if (wave < 4) WAIT_V(0);
    FS0(8);
    BARRIER();
    FS0(9);
    if (fwave == 0)
        asm volatile("s_mov_b64 exec, 0xff\n\tglobal_store_dword %0, %1, off sc0 sc1\n\ts_mov_b64 exec, 1\n\t"
                     "global_store_dword %2, %1, %3 sc1\n\ts_mov_b64 exec, -1" :: "v"(peer), "v"(callv), "v"(cntoff), "s"(myws) : "memory");
    GAS unsigned* tmo = (GAS unsigned*)(myws + WS_TMO);
    if (grp == 0 && ct != 0 && wave < 4) {
        const GAS unsigned* fl = (const GAS unsigned*)(myws + WS_FLAG1 + ((srcs >> (4 * ct)) & 7u) * 8192 + (panel & 1) * 4096 + (panel >> 1) * 8 + (wave >> 1) * 4);
        const unsigned long long t0 = __builtin_amdgcn_s_memrealtime();
        bool ok = true;
        for (unsigned it = 1; !__all(__hip_atomic_load(fl, __ATOMIC_RELAXED, __HIP_MEMORY_SCOPE_SYSTEM) >= call); ++it) {
            if ((it & 63u) == 0u
                && (__builtin_amdgcn_readfirstlane(__hip_atomic_load(tmo, __ATOMIC_RELAXED, __HIP_MEMORY_SCOPE_AGENT)) == call
                    || __builtin_amdgcn_s_memrealtime() - t0 > DEADLINE_TICKS)) {
                if (lane == 0) __hip_atomic_store(tmo, call, __ATOMIC_RELAXED, __HIP_MEMORY_SCOPE_AGENT);
                ok = false;
                break;
            }
            __builtin_amdgcn_s_sleep(1);
        }
        STAMP(5);
        if (ok) {
            u32x4 c0, c1, c2, c3;
            asm volatile("global_load_dwordx4 %0, %4, %5 sc0 sc1\n\tglobal_load_dwordx4 %1, %4, %5 offset:1024 sc0 sc1\n\t"
                         "global_load_dwordx4 %2, %4, %5 offset:2048 sc0 sc1\n\tglobal_load_dwordx4 %3, %4, %5 offset:3072 sc0 sc1\n\ts_waitcnt vmcnt(0)"
                         : "=&v"(c0), "=&v"(c1), "=&v"(c2), "=&v"(c3)
                         : "v"(lane16), "s"(wavews + stage_off + RAW0 + panel * A_STAGE + wave * 4096) : "memory");
            asm volatile("global_store_dwordx4 %4, %0, %5 sc0 sc1\n\tglobal_store_dwordx4 %4, %1, %5 offset:1024 sc0 sc1\n\t"
                         "global_store_dwordx4 %4, %2, %5 offset:2048 sc0 sc1\n\tglobal_store_dwordx4 %4, %3, %5 offset:3072 sc0 sc1\n\ts_nop 1"
                         :: "v"(c0), "v"(c1), "v"(c2), "v"(c3), "v"(lane16), "s"(myws + WS_PRE + (size_t)(ct * 4 + panel) * A_STAGE + wave * 4096) : "memory");
        }
        WAIT_V(0);
        if (lane == 0) __hip_atomic_store((GAS unsigned*)(myws + WS_FLAG2 + ct * 4096 + (wave * 4 + panel) * 128), call, __ATOMIC_RELAXED, __HIP_MEMORY_SCOPE_SYSTEM);
    }
    if (grp == 0) BARRIER();
    if (wave >= 4) {
        __builtin_amdgcn_s_sleep(AG_BDLY);
        STAGE_B2X(1);
        STAGE_B2X(2);
        STAGE_B2X(3);
        WAIT_V(0);
    }
    STAMP(1); FS0(10);

    const int wm = wave >> 2, wn = wave & 3;
    unsigned brd = (16 * g + (r16 >> 1)) * 256 + (((wn * 4) ^ ((r16 >> 1) | ((g & 1) << 3))) << 4) + (lane & 1) * 8;
#define B_PIN() asm volatile("" : "+v"(brd))
#define READ_BH(dst, n, h, so) do { \
        LAS unsigned char* bp_ = (LAS unsigned char*)(size_t)((brd ^ ((n) << 4)) + (lds0 + LDS_B + (so))); \
        const i32x2 t0_ = __builtin_amdgcn_ds_read_tr8_b64_v2i32((LAS i32x2*)(bp_ + (h) * 16384)); \
        const i32x2 t1_ = __builtin_amdgcn_ds_read_tr8_b64_v2i32((LAS i32x2*)(bp_ + (h) * 16384 + 2048)); \
        dst[n][4 * (h)] = t0_[0]; dst[n][4 * (h) + 1] = t0_[1]; dst[n][4 * (h) + 2] = t1_[0]; dst[n][4 * (h) + 3] = t1_[1]; } while (0)
#define READ_B1(dst, n, so) do { READ_BH(dst, n, 0, so); READ_BH(dst, n, 1, so); } while (0)
#define READ_B(dst, t) do { const int so_ = ((t) & 3) * B_STAGE; B_PIN(); READ_B1(dst, 0, so_); READ_B1(dst, 1, so_); READ_B1(dst, 2, so_); READ_B1(dst, 3, so_); } while (0)
    i32x8 bf0[4], bf1[4], bf2[4], bf3[4], a3e, a3o;
    volatile LAS unsigned* ctrl = (volatile LAS unsigned*)(lds + LDS_A);
    unsigned long long t1 = 0;
    if (wave != 0) {
        BARRIER();
    } else {
        const unsigned char* fl = myws + WS_FLAG1 + (size_t)src_rank * 8192 + lane * 128;
        const unsigned long long t0 = __builtin_amdgcn_s_memrealtime();
        unsigned bad = 0;
        u32x4 f0, f1;
#define POLL(f) asm volatile("global_load_dwordx4 %0, %1, off sc0 sc1" : "=&v"(f) : "v"(fl) : "memory")
#define SEEN(f) __all(f[0] >= call && f[1] >= call && f[2] >= call && f[3] >= call)
        POLL(f0);
        for (unsigned it = 1;; ++it) {
            __builtin_amdgcn_s_sleep(2);
            POLL(f1); asm volatile("s_waitcnt vmcnt(1)" : "+v"(f0) :: "memory");
            if (SEEN(f0)) break;
            __builtin_amdgcn_s_sleep(2);
            POLL(f0); asm volatile("s_waitcnt vmcnt(1)" : "+v"(f1) :: "memory");
            if (SEEN(f1)) break;
            if ((it & 63u) == 0u
                && (__builtin_amdgcn_readfirstlane(__hip_atomic_load(tmo, __ATOMIC_RELAXED, __HIP_MEMORY_SCOPE_AGENT)) == call
                    || __builtin_amdgcn_s_memrealtime() - t0 > DEADLINE_TICKS)) {
                if (lane == 0) __hip_atomic_store(tmo, call, __ATOMIC_RELAXED, __HIP_MEMORY_SCOPE_AGENT);
                bad = 1;
                break;
            }
        }
        BARRIER();
        t1 = __builtin_amdgcn_s_memrealtime();
        asm volatile("s_waitcnt vmcnt(0)" : "+v"(f0), "+v"(f1) :: "memory");
        STAMP(2); WAIT_L(0);
        if (lane == 0) ctrl[0] = bad;
        if (grp != 0) {
            while (__builtin_amdgcn_s_memrealtime() - t1 < (unsigned long long)AG_PRE) {}
            __builtin_amdgcn_fence(__ATOMIC_ACQUIRE, "");
        }
    }
    READ_B(bf3, 0);
    READ_B(bf1, 1);
    if (wave == 0 && grp != 0) {
        WAIT_V(0);
        STAMP(3); WAIT_L(0);
        while (__builtin_amdgcn_s_memrealtime() - t1 < (unsigned long long)(panel == 0 ? AG_HOLD : AG_INV + panel * AG_PH)) __builtin_amdgcn_s_sleep(1);
    }
    WAIT_L(0);
    BARRIER();
    STAMP(4); FS0(11);
    const bool bad = __builtin_amdgcn_readfirstlane(ctrl[0]) != 0u;
    float* obase = outp + (size_t)(src_rank * MS + panel * BM + wm * 64) * NL + ct * BN + wn * 64;
#define STORE_OUT1(m, i, np, v) do { const f32x4 o_ = (v); \
        asm volatile("global_store_dwordx4 %0, %1, %2 offset:%3 sc1\n\ts_nop 1" :: "v"(ooff), "v"(o_), "s"(obase + (size_t)((m) * 16 + (i) * 8) * NL), "i"((np) * 128) : "memory"); } while (0)

    if (bad) {
        const unsigned ooff = (unsigned)(((lane >> 3) * NL + (lane & 7) * 4) * 4);
        const float q = __builtin_nanf("");
#pragma unroll
        for (int m = 0; m < 4; ++m)
#pragma unroll
            for (int n = 0; n < 4; ++n) STORE_OUT1(m, n & 1, n >> 1, ((f32x4){q, q, q, q}));
        return;
    }

    struct Packed { u32x4 d0; u32x3 d1; unsigned e; } a0, a1;
    unsigned chk;
    LOAD_A(1, a1);
    STAGE_B(4);

    f32x4 acc[4][4];
#pragma unroll
    for (int m = 0; m < 4; ++m)
#pragma unroll
        for (int n = 0; n < 4; ++n) acc[m][n] = (f32x4){0.f, 0.f, 0.f, 0.f};
    const int awr = LDS_A + wave * 2048 + lane * 16;
    const int ard = LDS_A + wm * 8192 + lane * 16;
#define READ_A1(dst, slot, m) do { \
        const i32x4 lo_ = *(const LAS i32x4*)(lds + (slot) * A_STAGE + ard + (m) * 2048); \
        const i32x4 hi_ = *(const LAS i32x4*)(lds + (slot) * A_STAGE + ard + (m) * 2048 + 1024); \
        dst = (i32x8){lo_[0], lo_[1], lo_[2], lo_[3], hi_[0], hi_[1], hi_[2], hi_[3]}; } while (0)
#define READ_A(dst, slot) do { READ_A1(dst[0], slot, 0); READ_A1(dst[1], slot, 1); READ_A1(dst[2], slot, 2); READ_A1(dst[3], slot, 3); } while (0)
    unsigned hdr, b4, sp[7], o7, eg;
    const unsigned lane4 = (unsigned)lane * 0x01010101u | 0x80808080u;
#define PUT_A_H(r) do { asm volatile("" : "+v"(r.d0), "+v"(r.d1), "+v"(r.e)); \
        hdr = (unsigned)__builtin_amdgcn_readlane((int)r.e, ESC_CAP); \
        _Pragma("unroll") for (int i = 0; i < 4; ++i) sp[i] = r.d0[i] & 0x40404040u; \
        _Pragma("unroll") for (int i = 0; i < 3; ++i) sp[4 + i] = r.d1[i] & 0x40404040u; \
        b4 = (hdr & 0x7fu) * 0x01010101u; \
        asm volatile("" : "+v"(sp[0]), "+v"(sp[1]), "+v"(sp[2]), "+v"(sp[3]), "+v"(sp[4]), "+v"(sp[5]), "+v"(sp[6]), "+s"(b4)); } while (0)
#define PUT_A_0(r, slot) do { \
        *(LAS u32x4*)(lds + (slot) * A_STAGE + awr) = (u32x4){(r.d0[0] ^ sp[0]) + b4, (r.d0[1] ^ sp[1]) + b4, (r.d0[2] ^ sp[2]) + b4, (r.d0[3] ^ sp[3]) + b4}; \
        o7 = (((sp[1] << 1) | sp[0]) >> 6) + (((sp[3] << 1) | sp[2]) >> 4) + b4; asm volatile("" : "+v"(o7)); } while (0)
#define PUT_A_1(r, slot) \
        *(LAS u32x4*)(lds + (slot) * A_STAGE + awr + 1024) = (u32x4){(r.d1[0] ^ sp[4]) + b4, (r.d1[1] ^ sp[5]) + b4, (r.d1[2] ^ sp[6]) + b4, \
            o7 + (((sp[5] << 1) | sp[4]) >> 2) + (sp[6] << 1)}
#define LDS_XOR(off, v) __hip_atomic_fetch_xor((LAS unsigned*)(lds + (off)), (v), __ATOMIC_RELAXED, __HIP_MEMORY_SCOPE_WAVEFRONT)
#define PUT_A_G(r) do { \
        eg = (unsigned)__builtin_popcount((lane4 - (unsigned)__builtin_amdgcn_readlane((int)r.e, ESC_CAP + 1)) & 0x80808080u) \
           + (unsigned)__builtin_popcount((lane4 - (unsigned)__builtin_amdgcn_readlane((int)r.e, ESC_CAP + 5)) & 0x80808080u); \
        asm volatile("" : "+v"(eg)); } while (0)
#define PUT_A_2(r, slot, t) do { \
        if (eg < 8u) LDS_XOR(LDS_A + (slot) * A_STAGE + wave * 2048 + ((eg << 8) | (r.e & 0xfcu)), 0x40u << ((r.e & 3u) * 8u)); \
        if (__builtin_expect((hdr & 0x80u) != 0u, 0)) { unsigned m_; asm volatile("" : "+v"(lane16)); \
            asm volatile("global_load_dword %0, %1, %2 sc0 sc1\n\ts_waitcnt vmcnt(0)" : "=&v"(m_) : "v"(lane16 >> 2), "s"(aplane + (size_t)(t) * 2048) : "memory"); \
            _Pragma("unroll") for (int i = 0; i < 8; ++i) \
                LDS_XOR((slot) * A_STAGE + awr + (i >> 2) * 1024 + (i & 3) * 4, (i < 7 ? m_ << (6 - i) : m_ >> 1) & 0x40404040u); } } while (0)
#define PUT_A(r, slot, t) do { PUT_A_H(r); PUT_A_0(r, slot); PUT_A_1(r, slot); PUT_A_G(r); PUT_A_2(r, slot, t); } while (0)
#define MFMA(c, bfr, afr) c = __builtin_amdgcn_mfma_scale_f32_16x16x128_f8f6f4(bfr, afr, c, 1, 0, 0, 0, 0, 0)
#define MM(m, n, a, bf) do { SCHED(); MFMA(acc[m][n], bf[n], a); asm volatile("" : "+v"(acc[m][n])); SCHED(); } while (0)
#define MMA_ROW(m, a, bf) do { MM(m, 0, a, bf); MM(m, 1, a, bf); MM(m, 2, a, bf); MM(m, 3, a, bf); } while (0)
#define BODY(t, slot, bprev, bcur, bnxt, tnxt, a3cur, a3prev, rcur, rnxt, FIRST, DMA, tdma, PRE, WAIT, Y0, Y1, Y2, Y3, END, FINE) do { \
        FS(FINE, 0); \
        i32x8 f0_, f1_, f2_; \
        const int so_ = ((tnxt) & 3) * B_STAGE; \
        const unsigned char* bs_ = bbase + (size_t)(tdma) * (BK * NL); \
        const unsigned bd_ = lds0 + LDS_B + ((tdma) & 3) * B_STAGE + wave * 4096; \
        B_PIN(); READ_A1(f0_, slot, 0); \
        FS(FINE, 1); if (!(FIRST)) MM(3, 0, a3prev, bprev); \
        READ_A1(f1_, slot, 1); if ((DMA) == 2) STAGE_B1(2, bs_, bd_); \
        if (!(FIRST)) MM(3, 1, a3prev, bprev); \
        READ_B1(bnxt, 0, so_); \
        if (!(FIRST)) MM(3, 2, a3prev, bprev); \
        READ_BH(bnxt, 1, 0, so_); if (DMA) STAGE_B1(0, bs_, bd_); \
        if (!(FIRST)) MM(3, 3, a3prev, bprev); \
        READ_BH(bnxt, 1, 1, so_); \
        FS(FINE, 2); MM(0, 0, f0_, bcur); FS(FINE, 3); READ_A1(f2_, slot, 2); if ((DMA) == 2) STAGE_B1(3, bs_, bd_); \
        MM(0, 1, f0_, bcur); READ_B1(bnxt, 2, so_); \
        MM(0, 2, f0_, bcur); READ_A1(a3cur, slot, 3); \
        MM(0, 3, f0_, bcur); if (AG_B3) READ_B1(bnxt, 3, so_); PRE; FS(FINE, 4); if (DMA) STAGE_B1(1, bs_, bd_); \
        FS(FINE, 5); WAIT; FS(FINE, 6); if (AG_LP == 0) LOAD_A((t) + 2, rnxt); FS(FINE, 7); \
        MM(1, 0, f1_, bcur); PUT_A_H(rcur); if (AG_LP == 1) LOAD_A((t) + 2, rnxt); \
        MM(1, 1, f1_, bcur); PUT_A_0(rcur, 1 - (slot)); if (AG_LP == 2) LOAD_A((t) + 2, rnxt); \
        FS(FINE, 8); MM(1, 2, f1_, bcur); PUT_A_1(rcur, 1 - (slot)); if (AG_LP == 3) LOAD_A((t) + 2, rnxt); \
        MM(1, 3, f1_, bcur); if (!AG_B3) READ_BH(bnxt, 3, 0, so_); PUT_A_G(rcur); Y0; \
        FS(FINE, 9); MM(2, 0, f2_, bcur); PUT_A_2(rcur, 1 - (slot), (t) + 1); \
        MM(2, 1, f2_, bcur); if (!AG_B3) READ_BH(bnxt, 3, 1, so_); if ((DMA) == 1) STAGE_B1(2, bs_, bd_); Y1; \
        MM(2, 2, f2_, bcur); Y2; \
        MM(2, 3, f2_, bcur); if ((DMA) == 1) STAGE_B1(3, bs_, bd_); Y3; END; \
        FS(FINE, 10); WAIT_L(0); FS(FINE, 11); BARRIER(); FS(FINE, 12); STAMP(16 + (t)); STAMPL(t); } while (0)

    WAIT_V(4); LOAD_A(2, a0); PUT_A(a1, 1, 1); WAIT_L(0); BARRIER();
    STAMP(6);
    BODY(1, 1, bf0, bf1, bf0, 2, a3o, a3e, a0, a1, true, 1, 5, {}, WAIT_V(2), {}, {}, {}, {}, {}, 0);
    const unsigned char* xsrc = (grp == 0 ? myws + stage_off + RAW0 : myws + WS_PRE + grp * 4 * A_STAGE) + panel * A_STAGE + wm * 8192;
    i32x4 xlo, xhi;
#define MFMA_TXT(d, bb) "v_mfma_f32_16x16x128_f8f6f4 %" #d ", %" #bb ", %8, %" #d " cbsz:1\n\t"
#define MMA_X(n) do { SCHED(); \
        asm volatile("s_cmp_lt_u32 %11, 14\n\ts_cbranch_scc0 2f\n\ts_cmp_lt_u32 %11, 10\n\ts_cbranch_scc0 1f\n\t" \
                     MFMA_TXT(0, 9) MFMA_TXT(1, 10) "s_branch 4f\n1:\n\t" MFMA_TXT(2, 9) MFMA_TXT(3, 10) "s_branch 4f\n" \
                     "2:\n\ts_cmp_lt_u32 %11, 18\n\ts_cbranch_scc0 3f\n\t" MFMA_TXT(4, 9) MFMA_TXT(5, 10) "s_branch 4f\n3:\n\t" \
                     MFMA_TXT(6, 9) MFMA_TXT(7, 10) "\n4:" \
                     : "+v"(acc[0][n]), "+v"(acc[0][(n) + 1]), "+v"(acc[1][n]), "+v"(acc[1][(n) + 1]), \
                       "+v"(acc[2][n]), "+v"(acc[2][(n) + 1]), "+v"(acc[3][n]), "+v"(acc[3][(n) + 1]) \
                     : "v"(ax), "v"(bf3[n]), "v"(bf3[(n) + 1]), "s"(t) : "scc"); SCHED(); } while (0)
#define LOAD_X(r, off) asm volatile("global_load_dwordx4 %0, %1, %2 sc0 sc1" : "=&v"(r) : "v"(lane16), "s"(xsrc + (off)) : "memory")
#define EVEN(t, PRE, Y) BODY(t, 0, bf1, bf0, bf1, (t) + 1, a3e, a3o, a1, a0, false, 1, (t) + 4, PRE, WAIT_V(4), {}, {}, Y, {}, {}, 0)
#define ODD(t, PRE, Y) BODY(t, 1, bf0, bf1, bf0, (t) + 1, a3o, a3e, a0, a1, false, 1, (t) + 4, PRE, WAIT_V(4), {}, {}, Y, {}, {}, (__builtin_constant_p(t) && (t) == 5))
    EVEN(2, asm volatile("s_cmp_lg_u32 %3, 0\n\ts_cbranch_scc1 1f\n\tglobal_load_dword %0, %1, %2 sc0 sc1\n1:"
                         : "=&v"(chk) : "v"((lane16 & 0xf0u) * 8u), "s"(myws + WS_FLAG2 + grp * 4096), "s"(wave) : "memory", "scc"), {});
    ODD(3, {}, {});
    EVEN(4, {
        asm volatile("" : "+v"(chk));
        if (__builtin_expect(wave == 0 && grp != 0 && !__all(chk >= call), 0)) {
            const GAS unsigned* fl = (const GAS unsigned*)(myws + WS_FLAG2 + grp * 4096 + (lane16 & 0xf0u) * 8u);
            const unsigned long long t0 = __builtin_amdgcn_s_memrealtime();
            while (!__all(__hip_atomic_load(fl, __ATOMIC_RELAXED, __HIP_MEMORY_SCOPE_SYSTEM) >= call)) {
                if (__builtin_amdgcn_readfirstlane(__hip_atomic_load(tmo, __ATOMIC_RELAXED, __HIP_MEMORY_SCOPE_AGENT)) == call
                    || __builtin_amdgcn_s_memrealtime() - t0 > DEADLINE_TICKS) {
                    if (lane16 == 0) __hip_atomic_store(tmo, call, __ATOMIC_RELAXED, __HIP_MEMORY_SCOPE_AGENT);
                    break;
                }
                __builtin_amdgcn_s_sleep(1);
            }
            WAIT_V(0);
        } }, {});
    ODD(5, asm volatile("global_load_dword %0, %1, %2 sc1" : "=&v"(chk) : "v"(0u), "s"(tmo) : "memory"), {});
    for (int t = 6; t < NT - 10; t += 4) {
        i32x8 ax;
        EVEN(t, {}, LOAD_X(xlo, (t - 6) * 512));
        ODD(t + 1, LOAD_X(xhi, (t - 6) * 512 + 1024), {});
        EVEN(t + 2, {}, { asm volatile("" : "+v"(xlo), "+v"(xhi)); ax = __builtin_shufflevector(xlo, xhi, 0, 1, 2, 3, 4, 5, 6, 7); MMA_X(0); });
        ODD(t + 3, {}, MMA_X(2));
    }
    asm volatile("" : "+v"(chk));
    unsigned scale_bits = __builtin_amdgcn_readfirstlane(chk) == call ? 0x7fc00000u : scale0;
    asm volatile("" : "+s"(scale_bits));
    const float scale = __builtin_bit_cast(float, scale_bits);
#define STAGE_BP(j, t) STAGE_B1(j, bbase + (size_t)(t) * (BK * NL), lds0 + LDS_B + ((t) & 3) * B_STAGE + wave * 4096)
    BODY(22, 0, bf1, bf0, bf1, 23, a3e, a3o, a1, a0, false, 2, 26, {}, WAIT_V(6), {}, {}, {}, {}, {}, 0);
    BODY(23, 1, bf0, bf1, bf0, 24, a3o, a3e, a0, a1, false, 2, 27, {}, WAIT_V(4), {}, {}, {}, {}, {}, 0);
    BODY(24, 0, bf1, bf0, bf1, 25, a3e, a3o, a1, a0, false, 2, 28, READ_B1(bf2, 0, 2 * B_STAGE), WAIT_V(4),
         READ_B1(bf2, 1, 2 * B_STAGE), READ_B1(bf2, 2, 2 * B_STAGE), READ_B1(bf2, 3, 2 * B_STAGE), {}, {}, 0);
    BODY(25, 1, bf0, bf1, bf0, 27, a3o, a3e, a0, a1, false, 2, 29, {}, WAIT_V(4),
         STAGE_BP(0, 30), STAGE_BP(1, 30), STAGE_BP(2, 30), STAGE_BP(3, 30), {}, 0);
    BODY(26, 0, bf1, bf2, bf1, 28, a3e, a3o, a1, a0, false, 0, 0, {}, WAIT_V(4),
         STAGE_BP(0, 31), STAGE_BP(1, 31), STAGE_BP(2, 31), STAGE_BP(3, 31), WAIT_V(7), 0);
    {
        i32x8 f0_, f1_, f2_;
        B_PIN(); READ_A1(f0_, 1, 0);
        MM(3, 0, a3e, bf2); READ_A1(f1_, 1, 1);
        MM(3, 1, a3e, bf2); READ_B1(bf2, 0, B_STAGE);
        MM(3, 2, a3e, bf2); READ_B1(bf2, 1, B_STAGE);
        MM(3, 3, a3e, bf2); READ_A1(f2_, 1, 2);
        MM(0, 0, f0_, bf0); READ_BH(bf2, 2, 0, B_STAGE);
        MM(0, 1, f0_, bf0); READ_BH(bf2, 2, 1, B_STAGE);
        MM(0, 2, f0_, bf0); READ_BH(bf2, 3, 0, B_STAGE);
        MM(0, 3, f0_, bf0); READ_BH(bf2, 3, 1, B_STAGE);
        MM(1, 0, f1_, bf0); READ_A1(a3o, 1, 3);
        MM(1, 1, f1_, bf0); READ_BH(bf3, 0, 0, 2 * B_STAGE);
        MM(1, 2, f1_, bf0); READ_BH(bf3, 0, 1, 2 * B_STAGE);
        MM(1, 3, f1_, bf0); READ_BH(bf3, 1, 0, 2 * B_STAGE);
        MM(2, 0, f2_, bf0); READ_BH(bf3, 1, 1, 2 * B_STAGE);
        MM(2, 1, f2_, bf0); READ_BH(bf3, 2, 0, 2 * B_STAGE);
        MM(2, 2, f2_, bf0); READ_BH(bf3, 2, 1, 2 * B_STAGE);
        MM(2, 3, f2_, bf0); READ_BH(bf3, 3, 0, 2 * B_STAGE);
        MM(3, 0, a3o, bf0); READ_BH(bf3, 3, 1, 2 * B_STAGE);
        MM(3, 1, a3o, bf0); MM(3, 2, a3o, bf0); MM(3, 3, a3o, bf0);
        WAIT_V(4); LOAD_A(29, a1); PUT_A(a0, 0, 28);
        WAIT_V(3); WAIT_L(0); BARRIER(); STAMP(16 + 27);
    }
    asm volatile("" : "+v"(lane16));
    const unsigned ooff = (lane16 >> 7) * (NL * 4) + (lane16 & 0x70u);
    const unsigned twr = (lane16 & 0xf0u) * 8 + (((lane16 >> 4) ^ lane16) & 0x70u);
    const LAS unsigned char* trd = lds + LDS_B + wave * 4096 + (lane16 & 0x380u) + ((lane16 ^ (lane16 >> 3)) & 0x70u);
#define TAIL_PUT(m, n) *(LAS f32x4*)(lds + LDS_B + wave * 4096 + ((n) >> 1) * 2048 + (twr ^ (((n) & 1) * 64))) = acc[m][n]
#define TAIL_GET(v, np) do { v[0] = *(const LAS f32x4*)(trd + (np) * 2048); v[1] = *(const LAS f32x4*)(trd + (np) * 2048 + 1024); } while (0)
#define TAIL_OUT1(m, i, np, v) do { float e0_ = v[0] * scale, e1_ = v[1] * scale, e2_ = v[2] * scale, e3_ = v[3] * scale; \
        asm volatile("" : "+v"(e0_), "+v"(e1_), "+v"(e2_), "+v"(e3_)); \
        STORE_OUT1(m, i, np, ((f32x4){e0_, e1_, e2_, e3_})); } while (0)
#define TAIL(m, slot) do { \
        i32x8 af[4]; f32x4 u_[2], v_[2]; \
        READ_A(af, slot); \
        MM(m, 0, af[0], bf1); MM(m, 1, af[0], bf1); MM(m, 0, af[1], bf2); MM(m, 1, af[1], bf2); \
        MM(m, 0, af[2], bf3); MM(m, 1, af[2], bf3); MM(m, 0, af[3], bf0); MM(m, 1, af[3], bf0); \
        FS((m) == 2, 14); \
        MM(m, 2, af[0], bf1); MM(m, 2, af[1], bf2); MM(m, 2, af[2], bf3); TAIL_PUT(m, 0); \
        MM(m, 2, af[3], bf0); TAIL_PUT(m, 1); TAIL_GET(u_, 0); \
        MM(m, 3, af[0], bf1); MM(m, 3, af[1], bf2); MM(m, 3, af[2], bf3); MM(m, 3, af[3], bf0); \
        FS((m) == 2, 15); \
        TAIL_PUT(m, 2); TAIL_OUT1(m, 0, 0, u_[0]); SCHED(); TAIL_PUT(m, 3); TAIL_GET(v_, 1); SCHED(); \
        TAIL_OUT1(m, 1, 0, u_[1]); TAIL_OUT1(m, 0, 1, v_[0]); TAIL_OUT1(m, 1, 1, v_[1]); } while (0)
    {
        i32x8 af[4]; f32x4 u_[2], v_[2];
        B_PIN();
        READ_A(af, 0);
        MM(0, 0, af[0], bf1); MM(0, 1, af[0], bf1); READ_B1(bf0, 0, 3 * B_STAGE); MM(0, 2, af[0], bf1); MM(0, 3, af[0], bf1); READ_B1(bf0, 1, 3 * B_STAGE);
        MM(0, 0, af[1], bf2); MM(0, 1, af[1], bf2); READ_B1(bf0, 2, 3 * B_STAGE); MM(0, 2, af[1], bf2); MM(0, 3, af[1], bf2); READ_B1(bf0, 3, 3 * B_STAGE);
        MM(0, 0, af[2], bf3); MM(0, 1, af[2], bf3); MM(0, 0, af[3], bf0); MM(0, 1, af[3], bf0);
        MM(0, 2, af[2], bf3); MM(0, 2, af[3], bf0); MM(0, 3, af[2], bf3); TAIL_PUT(0, 0); MM(0, 3, af[3], bf0);
        TAIL_PUT(0, 1); TAIL_GET(u_, 0); TAIL_PUT(0, 2); TAIL_PUT(0, 3); TAIL_GET(v_, 1); SCHED();
        TAIL_OUT1(0, 0, 0, u_[0]); TAIL_OUT1(0, 1, 0, u_[1]); TAIL_OUT1(0, 0, 1, v_[0]); TAIL_OUT1(0, 1, 1, v_[1]);
    }
    WAIT_V(4); LOAD_A(30, a0); PUT_A(a1, 1, 29); WAIT_L(0); BARRIER();
    STAMP(8);
    TAIL(1, 1);
    WAIT_V(4); LOAD_A(31, a1); PUT_A(a0, 0, 30); WAIT_L(0); BARRIER();
    STAMP(9);
    FS(1, 13);
    TAIL(2, 0);
    FS(1, 16); WAIT_V(4); FS(1, 17); PUT_A(a1, 1, 31); WAIT_L(0); FS(1, 18); BARRIER(); FS(1, 19);
    STAMP(10);
    TAIL(3, 1);
}

}

extern "C" void kernel_launch(int rank, int nranks, ncclComm_t comm,
                              hipStream_t stream,
                              void* const* d_in, const int* in_sizes, int n_in,
                              void* d_out, int out_size,
                              const int* in_dtypes, int out_dtype,
                              void* const* d_out_all,
                              void* const* d_ws_all, size_t ws_bytes) {
    (void)comm; (void)in_sizes; (void)n_in; (void)out_size; (void)in_dtypes; (void)out_dtype; (void)d_out_all;
    if (nranks != NR || (unsigned)rank >= (unsigned)NR || ws_bytes < WS_END) { fprintf(stderr, "unexpected configuration\n"); return; }
    static bool attr_set = false;
    if (!attr_set) {
        HIP_CHECK(hipFuncSetAttribute((const void*)ag_gemm_kernel, hipFuncAttributeMaxDynamicSharedMemorySize, LDS_BYTES));
        attr_set = true;
    }
    static const int side[NR] = {0, 1, 0, 0, 1, 1, 0, 1};
    Params p;
    p.x = (const unsigned char*)d_in[0];
    p.w = (const unsigned char*)d_in[1];
    p.sx = (const float*)d_in[2];
    p.sw = (const float*)d_in[3];
    p.out = (float*)d_out;
    int order[NR], n = 0;
    order[n++] = rank;
    for (int r = 0; r < NR; ++r) if (r != rank && side[r] == side[rank]) order[n++] = r;
    for (int r = 0; r < NR; ++r) if (side[r] != side[rank]) order[n++] = r;
    p.srcs = 0;
    for (int i = 0; i < NR; ++i) { p.wsr[i] = (unsigned char*)d_ws_all[order[i]]; p.srcs |= (unsigned)order[i] << (4 * i); }
    p.rank = rank;
    hipLaunchKernelGGL(ag_gemm_kernel, dim3(NWG), dim3(NTHR), LDS_BYTES, stream, p);
    HIP_CHECK(hipGetLastError());
}
```
